```python
import numpy as np

kernel_name = "gpu_gemm_rs_m4096_k4096_n16384_w8a8_dqrelu"


ABSMAX_RTOL = 0.001

COMPUTE_DTYPE = "int8"
EPILOGUE = "dequant_relu"
IN_QUANT = "int8"
OUT_QUANT = None
IN_DTYPES = {"x": "int8", "w_mat": "int8",
             "scale_x": "float32", "scale_w": "float32"}
WORLD_SIZE = 8

def setup_inputs():

    x = np.random.randint(-127, 128, size=(4096, 4096)).astype(np.int8)
    w_mat = np.random.randint(-127, 128, size=(4096, 16384)).astype(np.int8)
    scale_x = (0.02 * (1.0 + np.random.rand(1))).astype(np.float32)
    scale_w = (0.02 * (1.0 + np.random.rand(1))).astype(np.float32)
    return {"x": x, "w_mat": w_mat, "scale_x": scale_x, "scale_w": scale_w}

def reference(x, w_mat, scale_x, scale_w):
    acc = x.astype(np.float64) @ w_mat.astype(np.float64)
    y = (acc.astype(np.float32) * (scale_x * scale_w)).astype(np.float32)
    return np.maximum(y, 0.0).astype(np.float32)
```

```cpp
#include <hip/hip_runtime.h>
#include <rccl/rccl.h>
#include <cstdio>
#include <cstdint>

#define HIP_CHECK(expr) do { hipError_t _e = (expr); if (_e != hipSuccess) { \
    fprintf(stderr, "HIP error %s at %s:%d\n", hipGetErrorString(_e), __FILE__, __LINE__); \
    return; } } while (0)

namespace {

typedef unsigned u32;
typedef unsigned long long u64;
typedef unsigned u32x2 __attribute__((ext_vector_type(2)));
typedef unsigned u32x4 __attribute__((ext_vector_type(4)));
typedef int i32x4 __attribute__((ext_vector_type(4)));
typedef float f32x2 __attribute__((ext_vector_type(2)));
typedef float f32x4 __attribute__((ext_vector_type(4)));
#define LAS __attribute__((address_space(3)))
#define GAS __attribute__((address_space(1)))

constexpr int NR = 8;
constexpr int MM = 4096, KK = 4096, NN = 16384;
constexpr int KS = KK / NR;
constexpr int NC = NN / NR;
constexpr int NWG = 256, NTHR = 512;

constexpr size_t OFF_CNT = 0;
constexpr size_t OFF_TMO = 32768;
constexpr size_t OFF_INFLAG = 65536;
constexpr size_t OFF_OFLAG = 131072;
constexpr size_t OFF_DBG = 1u << 20;
constexpr size_t OFF_XS = 2u << 20;
constexpr size_t XS_BYTES = 16u << 20;
constexpr size_t OFF_WS = OFF_XS + 2 * XS_BYTES;
constexpr size_t WS_BYTES = 8u << 20;
constexpr size_t OFF_OSLOT = OFF_WS + 2 * WS_BYTES;
constexpr u32 OSLOT = 256 + 8192;
constexpr size_t WS_NEED = OFF_OSLOT + (size_t)NR * 16 * 8 * 4 * OSLOT;

constexpr int HTB = 16384;
constexpr int LDS_SNAP = 8 * HTB;
constexpr int LDS_BYTES = 8 * HTB + 1024;
constexpr u64 DEADLINE = 30000000ull;

#define LAG 2
struct Sched { u64 qof[3]; u64 typ; u64 rnd0, rnd1; };
constexpr Sched make_sched() {
    Sched s{{0, 0, 0}, 0, 0, 0};
    int q = 0;
    for (int i = 0; i < 8 + LAG; ++i) {
        for (int t = 0; t < 3; ++t) {
            int r = (t == 2) ? i - LAG : i;
            if (r < 0 || r >= 8) continue;
            s.qof[t] |= (u64)q << (8 * r);
            s.typ |= (u64)t << (2 * q);
            if (q < 16) s.rnd0 |= (u64)r << (4 * q); else s.rnd1 |= (u64)r << (4 * (q - 16));
            ++q;
        }
    }
    return s;
}
constexpr Sched SCH = make_sched();

struct Params {
    const char* x; const char* w; const float* sx; const float* sw;
    float* out;
    char* ws[NR];
    int rank;
};

__device__ __forceinline__ void st16_sys(void* p, u32x4 v) {
    asm volatile("global_store_dwordx4 %0, %1, off sc0 sc1\n\ts_nop 1" :: "v"(p), "v"(v) : "memory");
}
#define WAIT_V(n) asm volatile("s_waitcnt vmcnt(" #n ")" ::: "memory")
#define WAIT_L(n) asm volatile("s_waitcnt lgkmcnt(" #n ")" ::: "memory")
#define BAR __builtin_amdgcn_s_barrier()
#define SCHED __builtin_amdgcn_sched_barrier(0)

__device__ __forceinline__ u32 ld_sys32(const char* p) {
    return __hip_atomic_load((const u32*)p, __ATOMIC_RELAXED, __HIP_MEMORY_SCOPE_SYSTEM);
}
__device__ __forceinline__ bool timed_out(char* ws, u32 call) {
    return __hip_atomic_load((const u32*)(ws + OFF_TMO), __ATOMIC_RELAXED, __HIP_MEMORY_SCOPE_AGENT) == call;
}
__device__ __forceinline__ bool give_up(char* ws, u32 call, u64 t0) {
    if (timed_out(ws, call)) return true;
    if (__builtin_amdgcn_s_memrealtime() - t0 > DEADLINE) {
        __hip_atomic_store((u32*)(ws + OFF_TMO), call, __ATOMIC_RELAXED, __HIP_MEMORY_SCOPE_AGENT);
        return true;
    }
    return false;
}
__device__ __forceinline__ u32 wave_incl_scan(u32 v) {
    v += (u32)__builtin_amdgcn_update_dpp(0, (int)v, 0x111, 0xf, 0xf, false);
    v += (u32)__builtin_amdgcn_update_dpp(0, (int)v, 0x112, 0xf, 0xf, false);
    v += (u32)__builtin_amdgcn_update_dpp(0, (int)v, 0x114, 0xf, 0xf, false);
    v += (u32)__builtin_amdgcn_update_dpp(0, (int)v, 0x118, 0xf, 0xf, false);
    v += (u32)__builtin_amdgcn_update_dpp(0, (int)v, 0x142, 0xa, 0xf, false);
    v += (u32)__builtin_amdgcn_update_dpp(0, (int)v, 0x143, 0xc, 0xf, false);
    return v;
}

#define STAMP_PARAM
#define STAMP_ARG
#define STAMP_DECL
#define STAMP() do {} while (0)
#define STAMP_AT(k, val) do {} while (0)
#define STAMP_FLUSH(ws) do {} while (0)

__device__ __forceinline__ void piece_load(const Params& P, int q, int dst, int u, int wv, int lane, u32 (&L)[16]) {
    const int typ = (int)((SCH.typ >> (2 * q)) & 3);
    const int rnd = (int)(((q < 16 ? SCH.rnd0 : SCH.rnd1) >> (4 * (q & 15))) & 15);
    if (typ == 0) {
        const char* src = P.w + (size_t)(rnd * 64 + (lane >> 4) * 16) * NN + dst * NC + u * 256 + wv * 64 + (lane & 15) * 4;
#pragma unroll
        for (int r = 0; r < 16; ++r) L[r] = *(const u32*)(src + (size_t)r * NN);
    } else {
        const int rb = 2 * u + (typ - 1);
        const char* src = P.x + (size_t)(rb * 256 + wv * 64 + (lane >> 2)) * KS + rnd * 64 + (((lane & 3) ^ ((lane >> 5) << 1)) << 4);
#pragma unroll
        for (int f = 0; f < 4; ++f) {
            u32x4 v = *(const u32x4*)(src + (size_t)f * 16 * KS);
            L[4 * f] = v.x; L[4 * f + 1] = v.y; L[4 * f + 2] = v.z; L[4 * f + 3] = v.w;
        }
    }
}

__device__ __forceinline__ void pusher(const Params& P, int dst, int u, int wid, int lane, u32 (&L)[16], u32 call STAMP_PARAM) {
    const int grp = wid >> 2, wv = wid & 3;
    char* wsd = P.ws[dst];
    const u32 par = call & 1;
    const int rel = (P.rank - dst) & 7;
    u32* flag = (u32*)(wsd + OFF_INFLAG + (size_t)(P.rank * 8 + u) * 128) + wid;
#pragma unroll 1
    for (int it = 0; it < 12; ++it) {
        const int q = 2 * it + grp;
        const int typ = (int)((SCH.typ >> (2 * q)) & 3);
        const int rnd = (int)(((q < 16 ? SCH.rnd0 : SCH.rnd1) >> (4 * (q & 15))) & 15);
        u32x4 S[4];
        char* d0; int dstep1, dstep2;
        if (typ == 0) {
#pragma unroll
            for (int k = 0; k < 4; ++k) {
                const u32 a = L[4 * k], b = L[4 * k + 1], c = L[4 * k + 2], d = L[4 * k + 3];
                const u32 t0 = __builtin_amdgcn_perm(b, a, 0x05010400u), t1 = __builtin_amdgcn_perm(b, a, 0x07030602u);
                const u32 t2 = __builtin_amdgcn_perm(d, c, 0x05010400u), t3 = __builtin_amdgcn_perm(d, c, 0x07030602u);
                S[0][k] = __builtin_amdgcn_perm(t2, t0, 0x05040100u); S[1][k] = __builtin_amdgcn_perm(t2, t0, 0x07060302u);
                S[2][k] = __builtin_amdgcn_perm(t3, t1, 0x05040100u); S[3][k] = __builtin_amdgcn_perm(t3, t1, 0x07060302u);
            }
            d0 = wsd + OFF_WS + par * WS_BYTES + (size_t)(u * 64 + rnd * 8 + rel) * 16384 + wv * 2048;
            dstep1 = 1024; dstep2 = 8192;
        } else {
#pragma unroll
            for (int f = 0; f < 4; ++f) S[f] = (u32x4){L[4 * f], L[4 * f + 1], L[4 * f + 2], L[4 * f + 3]};
            d0 = wsd + OFF_XS + par * XS_BYTES + (size_t)((2 * u + typ - 1) * 64 + rnd * 8 + rel) * 16384 + wv * 4096;
            dstep1 = 1024; dstep2 = 2048;
        }
        d0 += lane * 16;
        st16_sys(d0, S[0]); st16_sys(d0 + dstep1, S[1]); st16_sys(d0 + dstep2, S[2]); st16_sys(d0 + dstep2 + dstep1, S[3]);
        if (it + 1 < 12) piece_load(P, q + 2, dst, u, wv, lane, L);
        WAIT_V(0);
        if (lane == 0) __hip_atomic_store(flag, call * 16 + (u32)it + 1, __ATOMIC_RELAXED, __HIP_MEMORY_SCOPE_SYSTEM);
        STAMP();
    }
}

__device__ __forceinline__ void expander(const Params& P, LAS char* lds, int src, int h, int cb, int wid, int lane, u32 call STAMP_PARAM) {
    char* ws = P.ws[P.rank];
    const int wt = (src * 16 + h * 8 + cb) * 8 + wid;
    const u64* flag = (const u64*)(ws + OFF_OFLAG + (size_t)wt * 128);
    const float sc = __builtin_fabsf(P.sx[0] * P.sw[0]);
    LAS char* scr = lds + wid * 16384;
    const int wr = wid >> 2, wc = wid & 3, fr = lane & 15, fq = lane >> 4;
    float* orow = P.out + (size_t)(h * 256 + wr * 64 + fq * 4) * NN + src * NC + cb * 256 + wc * 64 + fr * 4;
#pragma unroll 1
    for (int sg = 0; sg < 4; ++sg) {
        u64 fv;
        {
            const u64 t0 = __builtin_amdgcn_s_memrealtime();
            for (;;) {
                fv = __hip_atomic_load(flag + sg, __ATOMIC_RELAXED, __HIP_MEMORY_SCOPE_SYSTEM);
                if ((u32)__builtin_amdgcn_readfirstlane((int)(fv >> 32)) >= call) break;
                if (give_up(ws, call, t0)) break;
                __builtin_amdgcn_s_sleep(8);
            }
        }
        STAMP();
        u32 info = (u32)__builtin_amdgcn_readfirstlane((int)(u32)fv);
        if (timed_out(ws, call)) info = 2u << 16;
        const u32 mode = info >> 16;
        u32 n = info & 0xffffu; if (n > 2048u) n = 2048u;
        float* o = orow + (size_t)((sg >> 1) * 128 + (sg & 1) * 32) * NN;
        if (mode >= 2) {
            const float qn = __builtin_nanf("");
#pragma unroll
            for (int g = 0; g < 8; ++g) *(f32x4*)(o + (size_t)((g >> 2) * 16 + (g & 3)) * NN) = (f32x4){qn, qn, qn, qn};
            continue;
        }
        const char* slot = ws + OFF_OSLOT + (size_t)(wt * 4 + sg) * OSLOT;
        const char* pc = slot + 256 + lane * 16;
        u32 mask; u32x4 c0, c1, c2;
        asm volatile("global_load_dword %0, %4, off sc0 sc1\n\t"
                     "global_load_dwordx4 %1, %5, off sc0 sc1\n\t"
                     "global_load_dwordx4 %2, %5, off offset:1024 sc0 sc1\n\t"
                     "global_load_dwordx4 %3, %5, off offset:2048 sc0 sc1\n\t"
                     "s_waitcnt vmcnt(0)"
                     : "=&v"(mask), "=&v"(c0), "=&v"(c1), "=&v"(c2) : "v"(slot + lane * 4), "v"(pc) : "memory");
        if (mode == 0) {
            auto unpack = [&](u32x4 c, int q) {
                const u32 m21 = 0x1fffffu;
                LAS f32x2* d = (LAS f32x2*)(scr + q * 24);
                d[0] = (f32x2){(float)(c.x & m21) * sc, (float)(__builtin_amdgcn_alignbit(c.y, c.x, 21) & m21) * sc};
                d[1] = (f32x2){(float)((c.y >> 10) & m21) * sc, (float)(c.z & m21) * sc};
                d[2] = (f32x2){(float)(__builtin_amdgcn_alignbit(c.w, c.z, 21) & m21) * sc, (float)((c.w >> 10) & m21) * sc};
            };
            unpack(c0, lane); unpack(c1, 64 + lane); unpack(c2, 128 + lane);
#pragma unroll 1
            for (u32 it = 3; it * 384u < n; ++it) {
                u32x4 c;
                if (it * 64 + lane < 342u) {
                    asm volatile("global_load_dwordx4 %0, %1, off sc0 sc1\n\ts_waitcnt vmcnt(0)" : "=&v"(c) : "v"(pc + it * 1024) : "memory");
                    unpack(c, it * 64 + lane);
                }
            }
        } else {
            auto unraw = [&](u32x4 c, int q) {
                *(LAS f32x4*)(scr + q * 16) = (f32x4){(float)c.x * sc, (float)c.y * sc, (float)c.z * sc, (float)c.w * sc};
            };
            unraw(c0, lane); unraw(c1, 64 + lane); unraw(c2, 128 + lane);
#pragma unroll 1
            for (u32 it = 3; it * 256u < n; ++it) {
                u32x4 c;
                asm volatile("global_load_dwordx4 %0, %1, off sc0 sc1\n\ts_waitcnt vmcnt(0)" : "=&v"(c) : "v"(pc + it * 1024) : "memory");
                unraw(c, it * 64 + lane);
            }
        }
        u32 incl = wave_incl_scan((u32)__builtin_popcount(mask));
        if (incl > 2048u) incl = 2048u;
        u32 a = incl * 4;
        __builtin_amdgcn_wave_barrier();
        u32 v[32];
#pragma unroll
        for (int e = 31; e >= 0; --e) {
            const int t = __builtin_amdgcn_sbfe((int)mask, e, 1);
            a += (u32)t << 2;
            v[e] = *(LAS u32*)(scr + (a & 0x3fffu)) & (u32)t;
        }
#pragma unroll
        for (int g = 0; g < 8; ++g)
            *(u32x4*)(o + (size_t)((g >> 2) * 16 + (g & 3)) * NN) = (u32x4){v[4 * g], v[4 * g + 1], v[4 * g + 2], v[4 * g + 3]};
        __builtin_amdgcn_wave_barrier();
    }
}

__device__ __forceinline__ const char* pair_flag_addr(const char* ws, int rank, int Pn, int lane, int cb, int own) {
    const int s = (rank + (Pn & 1) * 4 + (lane >> 4)) & 7;
    const int u = (lane & 8) ? own : cb;
    return ws + OFF_INFLAG + (s * 8 + u) * 128 + (lane & 7) * 4;
}
__device__ __forceinline__ u32 pair_thr(int Pn, int lane, int h, u32 call) {
    const int i = Pn >> 1;
    const u32 qw = (u32)((SCH.qof[0] >> (8 * i)) & 255), qx = (u32)(((h ? SCH.qof[2] : SCH.qof[1]) >> (8 * i)) & 255);
    const u32 q = (lane & 8) ? qx : qw;
    return (((u32)(lane >> 2) & 1u) == (q & 1u)) ? call * 16 + (q >> 1) + 1 : 0u;
}

__device__ __forceinline__ void gemm_wg(const Params& P, LAS char* lds, int own, int h, int cb, int wid, int lane, u32 call) {
    char* ws = P.ws[P.rank];
    const int tid = threadIdx.x, wr = wid >> 2, wc = wid & 3, fr = lane & 15, fq = lane >> 4;
    const u32 par = call & 1;
    const char* cA = ws + OFF_XS + par * XS_BYTES + (size_t)(2 * own + h) * (64 * 16384);
    const char* cB = ws + OFF_WS + par * WS_BYTES + (size_t)cb * (64 * 16384);
    const u32 voff = (u32)tid * 16;
    const u32 ldsw = (u32)wid * 1024;
    const int aoff = wr * 4096 + fr * 64 + ((fq ^ ((fr >> 3) << 1)) << 4), boff = wc * 2048 + lane * 16;
    STAMP_DECL;
    STAMP();
#define SA(b, hh) (((b) * 2 + (hh)) * HTB)
#define SB(b, hh) ((4 + (b) * 2 + (hh)) * HTB)
#define STAGE(bufoff, gbase) do { _Pragma("unroll") for (int _i = 0; _i < 2; ++_i) \
        __builtin_amdgcn_global_load_lds((const GAS u32*)((gbase) + _i * 16384 + voff), (LAS u32*)(lds + (bufoff) + ldsw + _i * 8192), 16, 0, 17); } while (0)
#define SNAP(Pn) do { if (wid == 0) __builtin_amdgcn_global_load_lds((const GAS u32*)pair_flag_addr(ws, P.rank, (Pn), lane, cb, own), \
        (LAS u32*)(lds + LDS_SNAP + ((Pn) & 1) * 256), 4, 0, 17); } while (0)
#define LDA(dst, b, hh) do { _Pragma("unroll") for (int m = 0; m < 4; ++m) _Pragma("unroll") for (int k = 0; k < 2; ++k) \
        dst[m][k] = *(const LAS i32x4*)(lds + SA(b, hh) + aoff + m * 1024 + k * 8192); } while (0)
#define LDB(dst, b, hh) do { _Pragma("unroll") for (int n = 0; n < 2; ++n) _Pragma("unroll") for (int k = 0; k < 2; ++k) \
        dst[n][k] = *(const LAS i32x4*)(lds + SB(b, hh) + boff + n * 1024 + k * 8192); } while (0)
#define MMA(ai, bj, At, Bt) do { __builtin_amdgcn_s_setprio(1); _Pragma("unroll") for (int m = 0; m < 4; ++m) _Pragma("unroll") for (int n = 0; n < 2; ++n) \
        _Pragma("unroll") for (int k = 0; k < 2; ++k) acc[ai][bj][m][n] = __builtin_amdgcn_mfma_i32_16x16x64_i8(At[m][k], Bt[n][k], acc[ai][bj][m][n], 0, 0, 0); \
        __builtin_amdgcn_s_setprio(0); } while (0)
    i32x4 acc[2][2][4][2];
#pragma unroll
    for (int a = 0; a < 2; ++a)
#pragma unroll
        for (int b = 0; b < 2; ++b)
#pragma unroll
            for (int m = 0; m < 4; ++m)
#pragma unroll
                for (int n = 0; n < 2; ++n) acc[a][b][m][n] = (i32x4){0, 0, 0, 0};
    i32x4 At[4][2], B0[2][2], B1[2][2];
    int Pc = 0;
    bool pf = false;
#pragma unroll 1
    for (;;) {
        const char* a0 = cA + (size_t)Pc * 65536; const char* b0 = cB + (size_t)Pc * 65536;
        if (!pf) {
            if (wid == 0) {
                const char* fa = pair_flag_addr(ws, P.rank, Pc, lane, cb, own);
                const u32 thr = pair_thr(Pc, lane, h, call);
                const u64 t0 = __builtin_amdgcn_s_memrealtime();
                for (;;) {
                    if (__builtin_amdgcn_ballot_w64(ld_sys32(fa) < thr) == 0) break;
                    if (give_up(ws, call, t0)) break;
                    __builtin_amdgcn_s_sleep(2);
                }
            }
            asm volatile("" ::: "memory");
            if (wr == 0 || Pc == 0) BAR;
            asm volatile("" ::: "memory");
            STAMP();
            if (Pc < 15) SNAP(Pc + 1);
            STAGE(SB(0, 0), b0); STAGE(SA(0, 0), a0); STAGE(SB(0, 1), b0 + 8192); STAGE(SA(0, 1), a0 + 8192);
            if (wr == 1) BAR;
            WAIT_V(4); BAR;
            STAGE(SB(1, 0), b0 + 32768); STAGE(SA(1, 0), a0 + 32768); STAGE(SB(1, 1), b0 + 32768 + 8192);
            WAIT_V(6); BAR;
        }
        STAMP();
        pf = false;
        if (Pc < 15) {
            const u32 sv = *(const LAS u32*)(lds + LDS_SNAP + ((Pc + 1) & 1) * 256 + lane * 4);
            pf = __builtin_amdgcn_ballot_w64(sv < pair_thr(Pc + 1, lane, h, call)) == 0;
        }
        const char* a2 = a0 + 65536; const char* b2 = b0 + 65536;
        LDB(B0, 0, 0); SCHED; LDA(At, 0, 0); STAGE(SA(1, 1), a0 + 32768 + 8192);
        WAIT_L(8); BAR; WAIT_L(0); MMA(0, 0, At, B0); BAR; SCHED;
        LDB(B1, 0, 1); if (pf) STAGE(SB(0, 0), b2);
        BAR; WAIT_L(0); MMA(0, 1, At, B1); BAR;
        LDA(At, 0, 1); if (pf) STAGE(SA(0, 0), a2);
        BAR; WAIT_L(0); MMA(1, 0, At, B0); BAR; SCHED;
        if (pf) { STAGE(SB(0, 1), b2 + 8192); WAIT_V(6); } else { WAIT_V(0); }
        BAR; MMA(1, 1, At, B1); BAR;
        LDB(B0, 1, 0); SCHED; LDA(At, 1, 0);
        if (pf) { if (Pc < 14) SNAP(Pc + 2); STAGE(SA(0, 1), a2 + 8192); }
        WAIT_L(8); BAR; WAIT_L(0); MMA(0, 0, At, B0); BAR; SCHED;
        LDB(B1, 1, 1); if (pf) STAGE(SB(1, 0), b2 + 32768);
        BAR; WAIT_L(0); MMA(0, 1, At, B1); BAR;
        LDA(At, 1, 1); if (pf) STAGE(SA(1, 0), a2 + 32768);
        BAR; WAIT_L(0); MMA(1, 0, At, B0); BAR; SCHED;
        if (pf) { STAGE(SB(1, 1), b2 + 32768 + 8192); WAIT_V(6); }
        BAR; MMA(1, 1, At, B1); BAR;
        if (++Pc == 16) break;
    }
    if (wr == 0) BAR;
    STAMP();

    const bool dead = timed_out(ws, call);
    if (P.sx[0] * P.sw[0] < 0.f) {
#pragma unroll
        for (int a = 0; a < 2; ++a)
#pragma unroll
            for (int b = 0; b < 2; ++b)
#pragma unroll
                for (int m = 0; m < 4; ++m)
#pragma unroll
                    for (int n = 0; n < 2; ++n) acc[a][b][m][n] = -acc[a][b][m][n];
    }
    char* wso = P.ws[own];
    const int wt = (P.rank * 16 + h * 8 + cb) * 8 + wid;
    u64* oflag = (u64*)(wso + OFF_OFLAG + (size_t)wt * 128);
    LAS char* scr = lds + wid * 16384;
    u32 info_prev = 0;
#pragma unroll
    for (int sg = 0; sg < 4; ++sg) {
        int v[32];
#pragma unroll
        for (int e = 0; e < 32; ++e) v[e] = acc[sg >> 1][(e >> 1) & 1][(sg & 1) * 2 + (e >> 4)][e & 1][(e >> 2) & 3];
        u32 mask = 0; int mx = 0;
#pragma unroll
        for (int e = 0; e < 32; ++e) { mask |= (v[e] > 0 ? 1u : 0u) << e; mx = v[e] > mx ? v[e] : mx; }
        const u32 cnt = (u32)__builtin_popcount(mask);
        const u32 incl = wave_incl_scan(cnt);
        const u32 n = (u32)__builtin_amdgcn_readlane((int)incl, 63);
        const bool raw = __builtin_amdgcn_ballot_w64(mx >= (1 << 21)) != 0;
        u32 a = (incl - cnt) * 4;
        __builtin_amdgcn_wave_barrier();
#pragma unroll
        for (int e = 0; e < 32; ++e) if (v[e] > 0) { *(LAS int*)(scr + a) = v[e]; a += 4; }
        *(LAS u32*)(scr + 12288 + lane * 4) = mask;
        __builtin_amdgcn_wave_barrier();
        if (sg > 0) {
            WAIT_V(0);
            if (lane == 0) __hip_atomic_store(oflag + (sg - 1), ((u64)call << 32) | info_prev, __ATOMIC_RELAXED, __HIP_MEMORY_SCOPE_SYSTEM);
        }
        char* slot = wso + OFF_OSLOT + (size_t)(wt * 4 + sg) * OSLOT;
        if (lane < 16) st16_sys(slot + lane * 16, *(const LAS u32x4*)(scr + 12288 + lane * 16));
        if (!raw) {
            const u32 nch = (n + 5) / 6;
#pragma unroll 1
            for (u32 q = lane; q < nch; q += 64) {
                const LAS u32x2* s = (const LAS u32x2*)(scr + q * 24);
                const u32x2 p0 = s[0], p1 = s[1], p2 = s[2];
                u32x4 c;
                c.x = p0.x | (p0.y << 21); c.y = (p0.y >> 11) | (p1.x << 10);
                c.z = p1.y | (p2.x << 21); c.w = (p2.x >> 11) | (p2.y << 10);
                st16_sys(slot + 256 + q * 16, c);
            }
        } else {
            const u32 nch = (n + 3) / 4;
#pragma unroll 1
            for (u32 q = lane; q < nch; q += 64) st16_sys(slot + 256 + q * 16, *(const LAS u32x4*)(scr + q * 16));
        }
        info_prev = dead ? (2u << 16) : (n | (raw ? 1u << 16 : 0u));
        STAMP();
    }
    WAIT_V(0);
    if (lane == 0) __hip_atomic_store(oflag + 3, ((u64)call << 32) | info_prev, __ATOMIC_RELAXED, __HIP_MEMORY_SCOPE_SYSTEM);
    STAMP();
    STAMP_FLUSH(ws);
#undef SA
#undef SB
#undef STAGE
#undef SNAP
#undef LDA
#undef LDB
#undef MMA
}

__global__ __launch_bounds__(NTHR, 2) void fused_kernel(const Params P) {
    extern __shared__ __attribute__((aligned(16))) char smem[];
    LAS char* lds = (LAS char*)smem;
    const int wid = __builtin_amdgcn_readfirstlane((int)(threadIdx.x >> 6)), lane = threadIdx.x & 63;
    const int xcd = blockIdx.x & 7, j = blockIdx.x >> 3;
    char* ws = P.ws[P.rank];
    const bool is_pusher = j < 8;
    const int dst = (P.rank + j) & 7;
    u32 L[16];
    if (is_pusher) piece_load(P, wid >> 2, dst, xcd, wid & 3, lane, L);
    u32 call = 0;
    if (lane == 0) call = __hip_atomic_fetch_add((u32*)(ws + OFF_CNT + (size_t)blockIdx.x * 128) + wid, 1u, __ATOMIC_RELAXED, __HIP_MEMORY_SCOPE_AGENT) + 1;
    call = (u32)__builtin_amdgcn_readfirstlane((int)call);
    if (j >= 8 && j < 24) {
        const int g = j - 8;
        gemm_wg(P, lds, 2 * (xcd >> 1) + ((g >> 2) & 1), g >> 3, 4 * (xcd & 1) + (g & 3), wid, lane, call);
        return;
    }
    STAMP_DECL;
    STAMP();
    if (is_pusher) pusher(P, dst, xcd, wid, lane, L, call STAMP_ARG);
    expander(P, lds, j & 7, is_pusher ? 1 : 0, xcd, wid, lane, call STAMP_ARG);
    STAMP();
    STAMP_FLUSH(ws);
}

}

extern "C" void kernel_launch(int rank, int nranks, ncclComm_t comm, hipStream_t stream,
                              void* const* d_in, const int* in_sizes, int n_in,
                              void* d_out, int out_size, const int* in_dtypes, int out_dtype,
                              void* const* d_out_all, void* const* d_ws_all, size_t ws_bytes) {
    (void)comm; (void)in_dtypes; (void)out_dtype; (void)d_out_all;
    if (nranks != NR || (unsigned)rank >= (unsigned)NR || n_in != 4 || ws_bytes < WS_NEED ||
        in_sizes[0] != MM * KS || in_sizes[1] != KS * NN || out_size != (MM / NR) * NN) {
        fprintf(stderr, "kernel_launch: unexpected problem shape\n");
        return;
    }
    static bool attr_set[64] = {};
    int dev = 0;
    HIP_CHECK(hipGetDevice(&dev));
    if (!attr_set[dev & 63]) {
        HIP_CHECK(hipFuncSetAttribute((const void*)fused_kernel, hipFuncAttributeMaxDynamicSharedMemorySize, LDS_BYTES));
        attr_set[dev & 63] = true;
    }
    Params P;
    P.x = (const char*)d_in[0]; P.w = (const char*)d_in[1];
    P.sx = (const float*)d_in[2]; P.sw = (const float*)d_in[3];
    P.out = (float*)d_out;
    for (int p = 0; p < NR; ++p) P.ws[p] = (char*)d_ws_all[p];
    P.rank = rank;
    hipLaunchKernelGGL(fused_kernel, dim3(NWG), dim3(NTHR), LDS_BYTES, stream, P);
    HIP_CHECK(hipGetLastError());
}
```

```cpp
#include <hip/hip_runtime.h>
#include <rccl/rccl.h>
#include <cstdio>
#include <cstdint>

#define HIP_CHECK(expr) do { hipError_t _e = (expr); if (_e != hipSuccess) { \
    fprintf(stderr, "HIP error %s at %s:%d\n", hipGetErrorString(_e), __FILE__, __LINE__); \
    return; } } while (0)

namespace {

typedef unsigned u32;
typedef unsigned long long u64;
typedef unsigned u32x2 __attribute__((ext_vector_type(2)));
typedef unsigned u32x4 __attribute__((ext_vector_type(4)));
typedef int i32x4 __attribute__((ext_vector_type(4)));
typedef float f32x2 __attribute__((ext_vector_type(2)));
typedef float f32x4 __attribute__((ext_vector_type(4)));
#define LAS __attribute__((address_space(3)))
#define GAS __attribute__((address_space(1)))

constexpr int NR = 8;
constexpr int MM = 4096, KK = 4096, NN = 16384;
constexpr int KS = KK / NR;
constexpr int NC = NN / NR;
constexpr int NWG = 256, NTHR = 512;

constexpr size_t OFF_CNT = 0;
constexpr size_t OFF_TMO = 32768;
constexpr size_t OFF_CLAIM = 36864;
constexpr size_t OFF_GATE = 40960;
constexpr size_t OFF_INFLAG = 524288;
constexpr size_t OFF_OFLAG = 131072;
constexpr size_t OFF_DBG2 = 262144;
constexpr size_t OFF_DBG = 1u << 20;
constexpr size_t OFF_XS = 2u << 20;
constexpr size_t XS_BYTES = 16u << 20;
constexpr size_t OFF_WS = OFF_XS + 2 * XS_BYTES;
constexpr size_t WS_BYTES = 8u << 20;
constexpr size_t OFF_OSLOT = OFF_WS + 2 * WS_BYTES;
constexpr u32 OSLOT = 256 + 8192;
constexpr size_t WS_NEED = OFF_OSLOT + (size_t)NR * 16 * 8 * 4 * OSLOT;

constexpr int HTB = 16384;
constexpr int LDS_SNAP = 8 * HTB;
constexpr int LDS_BYTES = 8 * HTB + 1024;
constexpr u64 DEADLINE = 30000000ull;

#define LAG 3
struct Sched { u64 qof[3]; u64 typ; u64 rnd0, rnd1; };
constexpr Sched make_sched() {
    Sched s{{0, 0, 0}, 0, 0, 0};
    int q = 0;
    for (int i = 0; i < 8 + LAG; ++i) {
        for (int t = 0; t < 3; ++t) {
            int r = (t == 2) ? i - LAG : i;
            if (r < 0 || r >= 8) continue;
            s.qof[t] |= (u64)q << (8 * r);
            s.typ |= (u64)t << (2 * q);
            if (q < 16) s.rnd0 |= (u64)r << (4 * q); else s.rnd1 |= (u64)r << (4 * (q - 16));
            ++q;
        }
    }
    return s;
}
constexpr Sched SCH = make_sched();
#define NPW 6
#define GATE 740
#define MAP 1
#define GATE_ALL 0
#define OUTST 0
#define NOLOCAL 0
#define NOGEMM 0
#define PST "sc0 sc1"
#define NOFLAG 0
#define CONTIG 0
#define ESLEEP 32
#define XSLEEP 8
#define GSLEEP 8
constexpr u32 NUNIT = 768;

struct Params {
    const char* x; const char* w; const float* sx; const float* sw;
    float* out;
    char* ws[NR];
    int rank;
};

__device__ __forceinline__ void st16_sys(void* p, u32x4 v) {
    asm volatile("global_store_dwordx4 %0, %1, off " PST "\n\ts_nop 1" :: "v"(p), "v"(v) : "memory");
}
#define WAIT_V(n) asm volatile("s_waitcnt vmcnt(" #n ")" ::: "memory")
#define WAIT_L(n) asm volatile("s_waitcnt lgkmcnt(" #n ")" ::: "memory")
#define BAR __builtin_amdgcn_s_barrier()
#define SCHED __builtin_amdgcn_sched_barrier(0)

__device__ __forceinline__ u32 ld_sys32(const char* p) {
    return __hip_atomic_load((const u32*)p, __ATOMIC_RELAXED, __HIP_MEMORY_SCOPE_SYSTEM);
}
__device__ __forceinline__ bool timed_out(char* ws, u32 call) {
    return __hip_atomic_load((const u32*)(ws + OFF_TMO), __ATOMIC_RELAXED, __HIP_MEMORY_SCOPE_AGENT) == call;
}
__device__ __forceinline__ bool give_up(char* ws, u32 call, u64 t0, u32& pass) {
    if ((++pass & 255u) == 0 && timed_out(ws, call)) return true;
    if (__builtin_amdgcn_s_memrealtime() - t0 > DEADLINE) {
        __hip_atomic_store((u32*)(ws + OFF_TMO), call, __ATOMIC_RELAXED, __HIP_MEMORY_SCOPE_AGENT);
        return true;
    }
    return false;
}
template <int SLEEP> __device__ __forceinline__ void wait_words(char* ws, const char* p, u32 call) {
    const u64 t0 = __builtin_amdgcn_s_memrealtime();
    u32 pass = 0;
    while (__builtin_amdgcn_ballot_w64(ld_sys32(p) < call) != 0) {
        if (give_up(ws, call, t0, pass)) break;
        __builtin_amdgcn_s_sleep(SLEEP);
    }
}
__device__ __forceinline__ u32 wave_incl_scan(u32 v) {
    v += (u32)__builtin_amdgcn_update_dpp(0, (int)v, 0x111, 0xf, 0xf, false);
    v += (u32)__builtin_amdgcn_update_dpp(0, (int)v, 0x112, 0xf, 0xf, false);
    v += (u32)__builtin_amdgcn_update_dpp(0, (int)v, 0x114, 0xf, 0xf, false);
    v += (u32)__builtin_amdgcn_update_dpp(0, (int)v, 0x118, 0xf, 0xf, false);
    v += (u32)__builtin_amdgcn_update_dpp(0, (int)v, 0x142, 0xa, 0xf, false);
    v += (u32)__builtin_amdgcn_update_dpp(0, (int)v, 0x143, 0xc, 0xf, false);
    return v;
}

#define STAMP_PARAM
#define STAMP_ARG
#define STAMP_DECL
#define STAMP() do {} while (0)
#define STAMP_AT(k, val) do {} while (0)
#define STAMP_SET(k) do {} while (0)
#define STAMP_FLUSH(ws) do {} while (0)

__device__ __forceinline__ int unit_typ(u32 n) { return (int)((SCH.typ >> (2 * (n >> 5))) & 3); }
__device__ __forceinline__ int unit_rnd(u32 n) { return (int)(((n < 512 ? SCH.rnd0 : SCH.rnd1) >> (4 * ((n >> 5) & 15))) & 15); }
__device__ __forceinline__ const char* unit_src(const Params& P, u32 n, int dst) {
    const int u = (int)(n >> 2) & 7, wv = (int)n & 3, typ = unit_typ(n), rnd = unit_rnd(n);
    return typ == 0 ? P.w + (size_t)(rnd * 64) * NN + dst * NC + u * 256 + wv * 64
                    : P.x + (size_t)((2 * u + typ - 1) * 256 + wv * 64) * KS + rnd * 64;
}
#define UNIT_LOADS \
    "s_cmp_lg_u32 %[typ], 0\n\ts_cbranch_scc1 1f\n\t" \
    "global_load_dword v8, %[vw], %[sb]\n\tv_add_u32 %[tmp], 0x4000, %[vw]\n\t" \
    "global_load_dword v9, %[tmp], %[sb]\n\tv_add_u32 %[tmp], 0x4000, %[tmp]\n\t" \
    "global_load_dword v10, %[tmp], %[sb]\n\tv_add_u32 %[tmp], 0x4000, %[tmp]\n\t" \
    "global_load_dword v11, %[tmp], %[sb]\n\tv_add_u32 %[tmp], 0x4000, %[tmp]\n\t" \
    "global_load_dword v12, %[tmp], %[sb]\n\tv_add_u32 %[tmp], 0x4000, %[tmp]\n\t" \
    "global_load_dword v13, %[tmp], %[sb]\n\tv_add_u32 %[tmp], 0x4000, %[tmp]\n\t" \
    "global_load_dword v14, %[tmp], %[sb]\n\tv_add_u32 %[tmp], 0x4000, %[tmp]\n\t" \
    "global_load_dword v15, %[tmp], %[sb]\n\tv_add_u32 %[tmp], 0x4000, %[tmp]\n\t" \
    "global_load_dword v16, %[tmp], %[sb]\n\tv_add_u32 %[tmp], 0x4000, %[tmp]\n\t" \
    "global_load_dword v17, %[tmp], %[sb]\n\tv_add_u32 %[tmp], 0x4000, %[tmp]\n\t" \
    "global_load_dword v18, %[tmp], %[sb]\n\tv_add_u32 %[tmp], 0x4000, %[tmp]\n\t" \
    "global_load_dword v19, %[tmp], %[sb]\n\tv_add_u32 %[tmp], 0x4000, %[tmp]\n\t" \
    "global_load_dword v20, %[tmp], %[sb]\n\tv_add_u32 %[tmp], 0x4000, %[tmp]\n\t" \
    "global_load_dword v21, %[tmp], %[sb]\n\tv_add_u32 %[tmp], 0x4000, %[tmp]\n\t" \
    "global_load_dword v22, %[tmp], %[sb]\n\tv_add_u32 %[tmp], 0x4000, %[tmp]\n\t" \
    "global_load_dword v23, %[tmp], %[sb]\n\ts_branch 2f\n" \
    "1:\n\ts_cmp_eq_u32 %[typ], 3\n\ts_cbranch_scc1 2f\n\t" \
    "global_load_dwordx4 v[8:11], %[vx], %[sb]\n\tv_add_u32 %[tmp], 0x2000, %[vx]\n\t" \
    "global_load_dwordx4 v[12:15], %[tmp], %[sb]\n\tv_add_u32 %[tmp], 0x2000, %[tmp]\n\t" \
    "global_load_dwordx4 v[16:19], %[tmp], %[sb]\n\tv_add_u32 %[tmp], 0x2000, %[tmp]\n\t" \
    "global_load_dwordx4 v[20:23], %[tmp], %[sb]\n" \
    "2:\n\t"

__device__ __forceinline__ u32 pusher(const Params& P, int dst, int widx, int lane, u32* callcnt STAMP_PARAM) {
    char* wsd = P.ws[dst];
    const int rel = (P.rank - dst) & 7;
    u32* cnt = (u32*)(P.ws[P.rank] + OFF_CLAIM + (size_t)dst * 128);
    u32* flags = (u32*)(wsd + OFF_INFLAG + (size_t)P.rank * (24 * 8 * 128));
    u32* gate = (u32*)(P.ws[P.rank] + OFF_GATE + (size_t)(dst * 8 + (lane & 7)) * 128);
    const u32 vw = (u32)((lane >> 4) * 16 * NN + (lane & 15) * 4), vx = (u32)((lane >> 2) * KS + (((lane & 3) ^ ((lane >> 5) << 1)) << 4));
    u32 n = (u32)widx, n1 = (u32)widx + 8 * NPW;
    u32x4 L0, L1, L2, L3;
    u32 t, tmp;
#define FETCH(counter, nn) asm volatile("s_mov_b64 exec, 1\n\tglobal_atomic_add %[t], %[c], %[one], off sc0\n\ts_mov_b64 exec, -1\n\t" \
        UNIT_LOADS "s_waitcnt vmcnt(0)" \
        : "=&{v[8:11]}"(L0), "=&{v[12:15]}"(L1), "=&{v[16:19]}"(L2), "=&{v[20:23]}"(L3), [t] "=&v"(t), [tmp] "=&v"(tmp) \
        : [c] "v"(counter), [one] "v"(1u), [typ] "s"((nn) < NUNIT ? unit_typ(nn) : 3), [sb] "s"(unit_src(P, (nn) < NUNIT ? (nn) : 0u, dst)), \
          [vw] "v"(vw), [vx] "v"(vx) : "memory", "scc")
    FETCH(callcnt, n);
    const u32 call = (u32)__builtin_amdgcn_readfirstlane((int)t) + 1;
    STAMP();
    const u32 par = call & 1;
    const u32 cbase = (call - 1) * NUNIT - 2 * 8 * NPW;
#pragma unroll 1
    while (n < NUNIT) {
        const int u = (int)(n >> 2) & 7, wv = (int)n & 3, typ = unit_typ(n), rnd = unit_rnd(n);
        u32x4 S[4] = {L0, L1, L2, L3};
        char* d0; int dstep2;
        if (typ == 0) {
            const u32x4 T[4] = {L0, L1, L2, L3};
#pragma unroll
            for (int k = 0; k < 4; ++k) {
                const u32 a = T[k].x, b = T[k].y, c = T[k].z, d = T[k].w;
                const u32 t0 = __builtin_amdgcn_perm(b, a, 0x05010400u), t1 = __builtin_amdgcn_perm(b, a, 0x07030602u);
                const u32 t2 = __builtin_amdgcn_perm(d, c, 0x05010400u), t3 = __builtin_amdgcn_perm(d, c, 0x07030602u);
                S[0][k] = __builtin_amdgcn_perm(t2, t0, 0x05040100u); S[1][k] = __builtin_amdgcn_perm(t2, t0, 0x07060302u);
                S[2][k] = __builtin_amdgcn_perm(t3, t1, 0x05040100u); S[3][k] = __builtin_amdgcn_perm(t3, t1, 0x07060302u);
            }
            d0 = wsd + OFF_WS + par * WS_BYTES + (size_t)(u * 64 + rnd * 8 + rel) * 16384 + wv * 2048;
            dstep2 = 8192;
        } else {
            d0 = wsd + OFF_XS + par * XS_BYTES + (size_t)((2 * u + typ - 1) * 64 + rnd * 8 + rel) * 16384 + wv * 4096;
            dstep2 = 2048;
        }
        if (CONTIG) { d0 = wsd + OFF_XS + par * XS_BYTES + (size_t)(rel * NUNIT + n) * 4096; dstep2 = 2048; }
        d0 += lane * 16;
        if (!(NOLOCAL == 1 && rel == 0) && !(NOLOCAL == 2 && rel == 0 && typ != 0)) {
        st16_sys(d0, S[0]); st16_sys(d0 + 1024, S[1]); st16_sys(d0 + dstep2, S[2]); st16_sys(d0 + dstep2 + 1024, S[3]); }
        FETCH(cnt, n1);
        if (lane == 0 && !NOFLAG) __hip_atomic_store(flags + (n >> 2) * 32 + (n & 3), call, __ATOMIC_RELAXED, __HIP_MEMORY_SCOPE_SYSTEM);
        STAMP();
        n = n1;
        n1 = (u32)__builtin_amdgcn_readfirstlane((int)t) - cbase;
        if (n1 == GATE - 1 + 2 * 8 * NPW && lane < 8) __hip_atomic_store(gate, call, __ATOMIC_RELAXED, __HIP_MEMORY_SCOPE_SYSTEM);
    }
#undef FETCH
    return call;
}

__device__ __forceinline__ void expander(const Params& P, LAS char* lds, int src, int h, int cb, int wid, int lane, u32 call STAMP_PARAM) {
    char* ws = P.ws[P.rank];
    const int wt = (src * 16 + h * 8 + cb) * 8 + wid;
    const u64* flag = (const u64*)(ws + OFF_OFLAG + (size_t)wt * 128);
    const float sc = __builtin_fabsf(P.sx[0] * P.sw[0]);
    LAS char* scr = lds + wid * 16384;
    const int wr = wid >> 2, wc = wid & 3, fr = lane & 15, fq = lane >> 4;
    float* orow = P.out + (size_t)(h * 256 + wr * 64 + fq * 4) * NN + src * NC + cb * 256 + wc * 64 + fr * 4;
    if (wid == 0) wait_words<ESLEEP>(ws, ws + OFF_GATE + (size_t)((src == P.rank ? (src + 1) & 7 : src) * 8 + cb) * 128, call);
    asm volatile("" ::: "memory");
    BAR;
    asm volatile("" ::: "memory");
#pragma unroll 1
    for (int sg = 0; sg < 4; ++sg) {
        u64 fv;
        {
            const u64 t0 = __builtin_amdgcn_s_memrealtime();
            u32 pass = 0;
            for (;;) {
                fv = __hip_atomic_load(flag + sg, __ATOMIC_RELAXED, __HIP_MEMORY_SCOPE_SYSTEM);
                if ((u32)__builtin_amdgcn_readfirstlane((int)(fv >> 32)) >= call) break;
                if (give_up(ws, call, t0, pass)) break;
                __builtin_amdgcn_s_sleep(XSLEEP);
            }
        }
        STAMP();
        u32 info = (u32)__builtin_amdgcn_readfirstlane((int)(u32)fv);
        if (timed_out(ws, call)) info = 2u << 16;
        const u32 mode = info >> 16;
        u32 n = info & 0xffffu; if (n > 2048u) n = 2048u;
        float* o = orow + (size_t)((sg >> 1) * 128 + (sg & 1) * 32) * NN;
        if (mode >= 2) {
            const float qn = __builtin_nanf("");
#pragma unroll
            for (int g = 0; g < 8; ++g) *(f32x4*)(o + (size_t)((g >> 2) * 16 + (g & 3)) * NN) = (f32x4){qn, qn, qn, qn};
            continue;
        }
        const char* slot = ws + OFF_OSLOT + (size_t)(wt * 4 + sg) * OSLOT;
        const char* pc = slot + 256 + lane * 16;
        u32 mask; u32x4 c0, c1, c2;
        asm volatile("global_load_dword %0, %4, off sc0 sc1\n\t"
                     "global_load_dwordx4 %1, %5, off sc0 sc1\n\t"
                     "global_load_dwordx4 %2, %5, off offset:1024 sc0 sc1\n\t"
                     "global_load_dwordx4 %3, %5, off offset:2048 sc0 sc1\n\t"
                     "s_waitcnt vmcnt(0)"
                     : "=&v"(mask), "=&v"(c0), "=&v"(c1), "=&v"(c2) : "v"(slot + lane * 4), "v"(pc) : "memory");
        if (mode == 0) {
            auto unpack = [&](u32x4 c, int q) {
                const u32 m21 = 0x1fffffu;
                LAS f32x2* d = (LAS f32x2*)(scr + q * 24);
                d[0] = (f32x2){(float)(c.x & m21) * sc, (float)(__builtin_amdgcn_alignbit(c.y, c.x, 21) & m21) * sc};
                d[1] = (f32x2){(float)((c.y >> 10) & m21) * sc, (float)(c.z & m21) * sc};
                d[2] = (f32x2){(float)(__builtin_amdgcn_alignbit(c.w, c.z, 21) & m21) * sc, (float)((c.w >> 10) & m21) * sc};
            };
            unpack(c0, lane); unpack(c1, 64 + lane); unpack(c2, 128 + lane);
#pragma unroll 1
            for (u32 it = 3; it * 384u < n; ++it) {
                u32x4 c;
                if (it * 64 + lane < 342u) {
                    asm volatile("global_load_dwordx4 %0, %1, off sc0 sc1\n\ts_waitcnt vmcnt(0)" : "=&v"(c) : "v"(pc + it * 1024) : "memory");
                    unpack(c, it * 64 + lane);
                }
            }
        } else {
            auto unraw = [&](u32x4 c, int q) {
                *(LAS f32x4*)(scr + q * 16) = (f32x4){(float)c.x * sc, (float)c.y * sc, (float)c.z * sc, (float)c.w * sc};
            };
            unraw(c0, lane); unraw(c1, 64 + lane); unraw(c2, 128 + lane);
#pragma unroll 1
            for (u32 it = 3; it * 256u < n; ++it) {
                u32x4 c;
                asm volatile("global_load_dwordx4 %0, %1, off sc0 sc1\n\ts_waitcnt vmcnt(0)" : "=&v"(c) : "v"(pc + it * 1024) : "memory");
                unraw(c, it * 64 + lane);
            }
        }
        u32 incl = wave_incl_scan((u32)__builtin_popcount(mask));
        if (incl > 2048u) incl = 2048u;
        u32 a = incl * 4;
        __builtin_amdgcn_wave_barrier();
        u32 v[32];
#pragma unroll
        for (int e = 31; e >= 0; --e) {
            const int t = __builtin_amdgcn_sbfe((int)mask, e, 1);
            a += (u32)t << 2;
            v[e] = *(LAS u32*)(scr + (a & 0x3fffu)) & (u32)t;
        }
#pragma unroll
        for (int g = 0; g < 8; ++g) {
            const u32x4 ov = (u32x4){v[4 * g], v[4 * g + 1], v[4 * g + 2], v[4 * g + 3]};
            float* op = o + (size_t)((g >> 2) * 16 + (g & 3)) * NN;
            *(u32x4*)op = ov;
        }
        __builtin_amdgcn_wave_barrier();
    }
}

__device__ __forceinline__ const char* pair_flag_addr(const char* ws, int rank, int Pn, int lane, int cb, int own, int h) {
    const int s = (rank + (Pn & 1) * 4 + ((lane >> 3) & 3)) & 7, i = Pn >> 1;
    const int qw = (int)((SCH.qof[0] >> (8 * i)) & 255), qx = (int)(((h ? SCH.qof[2] : SCH.qof[1]) >> (8 * i)) & 255);
    return ws + OFF_INFLAG + ((s * 24 + ((lane & 4) ? qx : qw)) * 8 + ((lane & 4) ? own : cb)) * 128 + (lane & 3) * 4;
}

__device__ __forceinline__ void gemm_wg(const Params& P, LAS char* lds, int own, int h, int cb, int wid, int lane, u32 call STAMP_PARAM) {
    char* ws = P.ws[P.rank];
    const int tid = threadIdx.x, wr = wid >> 2, wc = wid & 3, fr = lane & 15, fq = lane >> 4;
    const u32 par = call & 1;
    const char* cA = ws + OFF_XS + par * XS_BYTES + (size_t)(2 * own + h) * (64 * 16384);
    const char* cB = ws + OFF_WS + par * WS_BYTES + (size_t)cb * (64 * 16384);
    const u32 voff = (u32)tid * 16;
    const u32 ldsw = (u32)wid * 1024;
    const int aoff = wr * 4096 + fr * 64 + ((fq ^ ((fr >> 3) << 1)) << 4), boff = wc * 2048 + lane * 16;
    STAMP();
#define SA(b, hh) (((b) * 2 + (hh)) * HTB)
#define SB(b, hh) ((4 + (b) * 2 + (hh)) * HTB)
#define STAGE(bufoff, gbase) do { _Pragma("unroll") for (int _i = 0; _i < 2; ++_i) \
        __builtin_amdgcn_global_load_lds((const GAS u32*)((gbase) + _i * 16384 + voff), (LAS u32*)(lds + (bufoff) + ldsw + _i * 8192), 16, 0, 17); } while (0)
#define SNAP(Pn) do { if (wid == 0) __builtin_amdgcn_global_load_lds((const GAS u32*)pair_flag_addr(ws, P.rank, (Pn), lane, cb, own, h), \
        (LAS u32*)(lds + LDS_SNAP + ((Pn) & 1) * 256), 4, 0, 17); } while (0)
#define LDA(dst, b, hh) do { _Pragma("unroll") for (int m = 0; m < 4; ++m) _Pragma("unroll") for (int k = 0; k < 2; ++k) \
        dst[m][k] = *(const LAS i32x4*)(lds + SA(b, hh) + aoff + m * 1024 + k * 8192); } while (0)
#define LDB(dst, b, hh) do { _Pragma("unroll") for (int n = 0; n < 2; ++n) _Pragma("unroll") for (int k = 0; k < 2; ++k) \
        dst[n][k] = *(const LAS i32x4*)(lds + SB(b, hh) + boff + n * 1024 + k * 8192); } while (0)
#define MMA(ai, bj, At, Bt) do { __builtin_amdgcn_s_setprio(1); _Pragma("unroll") for (int m = 0; m < 4; ++m) _Pragma("unroll") for (int n = 0; n < 2; ++n) \
        _Pragma("unroll") for (int k = 0; k < 2; ++k) acc[ai][bj][m][n] = __builtin_amdgcn_mfma_i32_16x16x64_i8(At[m][k], Bt[n][k], acc[ai][bj][m][n], 0, 0, 0); \
        __builtin_amdgcn_s_setprio(0); } while (0)
    i32x4 acc[2][2][4][2];
#pragma unroll
    for (int a = 0; a < 2; ++a)
#pragma unroll
        for (int b = 0; b < 2; ++b)
#pragma unroll
            for (int m = 0; m < 4; ++m)
#pragma unroll
                for (int n = 0; n < 2; ++n) acc[a][b][m][n] = (i32x4){0, 0, 0, 0};
    i32x4 At[4][2], B0[2][2], B1[2][2];
    int Pc = 0;
    bool pf = false;
#pragma unroll 1
    for (;;) {
        const char* a0 = cA + (size_t)Pc * 65536; const char* b0 = cB + (size_t)Pc * 65536;
        if (!pf) {
            if (wid == 0) wait_words<GSLEEP>(ws, pair_flag_addr(ws, P.rank, Pc, lane, cb, own, h), call);
            asm volatile("" ::: "memory");
            if (wr == 0 || Pc == 0) BAR;
            asm volatile("" ::: "memory");
            STAMP();
            if (Pc < 15) SNAP(Pc + 1);
            STAGE(SB(0, 0), b0); STAGE(SA(0, 0), a0); STAGE(SB(0, 1), b0 + 8192); STAGE(SA(0, 1), a0 + 8192);
            if (wr == 1) BAR;
            WAIT_V(4); BAR;
            STAGE(SB(1, 0), b0 + 32768); STAGE(SA(1, 0), a0 + 32768); STAGE(SB(1, 1), b0 + 32768 + 8192);
            WAIT_V(6); BAR;
        }
        STAMP();
        pf = false;
        if (Pc < 15) {
            const u32 sv = *(const LAS u32*)(lds + LDS_SNAP + ((Pc + 1) & 1) * 256 + lane * 4);
            pf = __builtin_amdgcn_ballot_w64(sv < call) == 0;
        }
        const char* a2 = a0 + 65536; const char* b2 = b0 + 65536;
        LDB(B0, 0, 0); SCHED; LDA(At, 0, 0); STAGE(SA(1, 1), a0 + 32768 + 8192);
        WAIT_L(8); BAR; WAIT_L(0); MMA(0, 0, At, B0); BAR; SCHED;
        LDB(B1, 0, 1); if (pf) STAGE(SB(0, 0), b2);
        BAR; WAIT_L(0); MMA(0, 1, At, B1); BAR;
        LDA(At, 0, 1); if (pf) STAGE(SA(0, 0), a2);
        BAR; WAIT_L(0); MMA(1, 0, At, B0); BAR; SCHED;
        if (pf) { STAGE(SB(0, 1), b2 + 8192); WAIT_V(6); } else { WAIT_V(0); }
        BAR; MMA(1, 1, At, B1); BAR;
        LDB(B0, 1, 0); SCHED; LDA(At, 1, 0);
        if (pf) { if (Pc < 14) SNAP(Pc + 2); STAGE(SA(0, 1), a2 + 8192); }
        WAIT_L(8); BAR; WAIT_L(0); MMA(0, 0, At, B0); BAR; SCHED;
        LDB(B1, 1, 1); if (pf) STAGE(SB(1, 0), b2 + 32768);
        BAR; WAIT_L(0); MMA(0, 1, At, B1); BAR;
        LDA(At, 1, 1); if (pf) STAGE(SA(1, 0), a2 + 32768);
        BAR; WAIT_L(0); MMA(1, 0, At, B0); BAR; SCHED;
        if (pf) { STAGE(SB(1, 1), b2 + 32768 + 8192); WAIT_V(6); }
        BAR; MMA(1, 1, At, B1); BAR;
        if (++Pc == 16) break;
    }
    if (wr == 0) BAR;
    STAMP_SET(40);
    STAMP();

    const bool dead = timed_out(ws, call);
    if (P.sx[0] * P.sw[0] < 0.f) {
#pragma unroll
        for (int a = 0; a < 2; ++a)
#pragma unroll
            for (int b = 0; b < 2; ++b)
#pragma unroll
                for (int m = 0; m < 4; ++m)
#pragma unroll
                    for (int n = 0; n < 2; ++n) acc[a][b][m][n] = -acc[a][b][m][n];
    }
    char* wso = P.ws[own];
    const int wt = (P.rank * 16 + h * 8 + cb) * 8 + wid;
    u64* oflag = (u64*)(wso + OFF_OFLAG + (size_t)wt * 128);
    LAS char* scr = lds + wid * 16384;
    u32 info_prev = 0;
#pragma unroll
    for (int sg = 0; sg < 4; ++sg) {
        int v[32];
#pragma unroll
        for (int e = 0; e < 32; ++e) v[e] = acc[sg >> 1][(e >> 1) & 1][(sg & 1) * 2 + (e >> 4)][e & 1][(e >> 2) & 3];
        u32 mask = 0; int mx = 0;
#pragma unroll
        for (int e = 0; e < 32; ++e) { mask |= (v[e] > 0 ? 1u : 0u) << e; mx = v[e] > mx ? v[e] : mx; }
        const u32 cnt = (u32)__builtin_popcount(mask);
        const u32 incl = wave_incl_scan(cnt);
        const u32 n = (u32)__builtin_amdgcn_readlane((int)incl, 63);
        const bool raw = __builtin_amdgcn_ballot_w64(mx >= (1 << 21)) != 0;
        u32 a = (incl - cnt) * 4;
        __builtin_amdgcn_wave_barrier();
#pragma unroll
        for (int e = 0; e < 32; ++e) if (v[e] > 0) { *(LAS int*)(scr + a) = v[e]; a += 4; }
        *(LAS u32*)(scr + 12288 + lane * 4) = mask;
        __builtin_amdgcn_wave_barrier();
        if (sg == 0) {
            if (wid == 0 && own != P.rank) wait_words<GSLEEP>(ws, ws + OFF_GATE + (size_t)((GATE_ALL ? (lane & 7) : own) * 8 + cb) * 128, call);
            asm volatile("" ::: "memory");
            BAR;
            asm volatile("" ::: "memory");
        }
        if (sg == 0) STAMP();
        if (sg > 0) {
            WAIT_V(0);
            if (lane == 0) __hip_atomic_store(oflag + (sg - 1), ((u64)call << 32) | info_prev, __ATOMIC_RELAXED, __HIP_MEMORY_SCOPE_SYSTEM);
        }
        char* slot = wso + OFF_OSLOT + (size_t)(wt * 4 + sg) * OSLOT;
        if (lane < 16) st16_sys(slot + lane * 16, *(const LAS u32x4*)(scr + 12288 + lane * 16));
        if (!raw) {
            const u32 nch = (n + 5) / 6;
#pragma unroll 1
            for (u32 q = lane; q < nch; q += 64) {
                const LAS u32x2* s = (const LAS u32x2*)(scr + q * 24);
                const u32x2 p0 = s[0], p1 = s[1], p2 = s[2];
                u32x4 c;
                c.x = p0.x | (p0.y << 21); c.y = (p0.y >> 11) | (p1.x << 10);
                c.z = p1.y | (p2.x << 21); c.w = (p2.x >> 11) | (p2.y << 10);
                st16_sys(slot + 256 + q * 16, c);
            }
        } else {
            const u32 nch = (n + 3) / 4;
#pragma unroll 1
            for (u32 q = lane; q < nch; q += 64) st16_sys(slot + 256 + q * 16, *(const LAS u32x4*)(scr + q * 16));
        }
        info_prev = dead ? (2u << 16) : (n | (raw ? 1u << 16 : 0u));
        STAMP();
    }
    WAIT_V(0);
    if (lane == 0) __hip_atomic_store(oflag + 3, ((u64)call << 32) | info_prev, __ATOMIC_RELAXED, __HIP_MEMORY_SCOPE_SYSTEM);
    STAMP();
    STAMP_FLUSH(ws);
#undef SA
#undef SB
#undef STAGE
#undef SNAP
#undef LDA
#undef LDB
#undef MMA
}

__global__ __launch_bounds__(NTHR, 2) void fused_kernel(const Params P) {
    extern __shared__ __attribute__((aligned(16))) char smem[];
    LAS char* lds = (LAS char*)smem;
    const int wid = __builtin_amdgcn_readfirstlane((int)(threadIdx.x >> 6)), lane = threadIdx.x & 63;
    const int xcd = blockIdx.x & 7, j = blockIdx.x >> 3;
    char* ws = P.ws[P.rank];
    const bool is_pusher = j < 8;
    const int dst = (P.rank + j) & 7;
    STAMP_DECL;
    STAMP();
    u32* callcnt = (u32*)(ws + OFF_CNT + (size_t)blockIdx.x * 128) + wid;
    u32 call = 0;
    if (is_pusher && wid < NPW) {
        call = pusher(P, dst, xcd * NPW + wid, lane, callcnt STAMP_ARG);
    } else {
        if (lane == 0) call = __hip_atomic_fetch_add(callcnt, 1u, __ATOMIC_RELAXED, __HIP_MEMORY_SCOPE_AGENT) + 1;
        call = (u32)__builtin_amdgcn_readfirstlane((int)call);
        if (j >= 8 && j < 24) {
            const int g = j - 8;
            if (NOGEMM) { STAMP_SET(40); for (int i = 0; i < 7; ++i) STAMP(); STAMP_FLUSH(ws); return; }
            gemm_wg(P, lds, g & 7, g >> 3, xcd, wid, lane, call STAMP_ARG);
            return;
        }
        STAMP();
    }
    STAMP_AT(40, __builtin_amdgcn_s_memrealtime()); STAMP_SET(41);
    if (NOGEMM) { for (int i = 0; i < 4; ++i) STAMP(); } else
    expander(P, lds, j & 7, is_pusher ? 1 : 0, xcd, wid, lane, call STAMP_ARG);
    STAMP();
    STAMP_FLUSH(ws);
}

}

extern "C" void kernel_launch(int rank, int nranks, ncclComm_t comm, hipStream_t stream,
                              void* const* d_in, const int* in_sizes, int n_in,
                              void* d_out, int out_size, const int* in_dtypes, int out_dtype,
                              void* const* d_out_all, void* const* d_ws_all, size_t ws_bytes) {
    (void)comm; (void)in_dtypes; (void)out_dtype; (void)d_out_all;
    if (nranks != NR || (unsigned)rank >= (unsigned)NR || n_in != 4 || ws_bytes < WS_NEED ||
        in_sizes[0] != MM * KS || in_sizes[1] != KS * NN || out_size != (MM / NR) * NN) {
        fprintf(stderr, "kernel_launch: unexpected problem shape\n");
        return;
    }
    static bool attr_set[64] = {};
    int dev = 0;
    HIP_CHECK(hipGetDevice(&dev));
    if (!attr_set[dev & 63]) {
        HIP_CHECK(hipFuncSetAttribute((const void*)fused_kernel, hipFuncAttributeMaxDynamicSharedMemorySize, LDS_BYTES));
        attr_set[dev & 63] = true;
    }
    Params P;
    P.x = (const char*)d_in[0]; P.w = (const char*)d_in[1];
    P.sx = (const float*)d_in[2]; P.sw = (const float*)d_in[3];
    P.out = (float*)d_out;
    for (int p = 0; p < NR; ++p) P.ws[p] = (char*)d_ws_all[p];
    P.rank = rank;
    hipLaunchKernelGGL(fused_kernel, dim3(NWG), dim3(NTHR), LDS_BYTES, stream, P);
    HIP_CHECK(hipGetLastError());
}
```

```cpp
#include <hip/hip_runtime.h>
#include <rccl/rccl.h>
#include <cstdio>
#include <cstdint>

#define HIP_CHECK(expr) do { hipError_t _e = (expr); if (_e != hipSuccess) { \
    fprintf(stderr, "HIP error %s at %s:%d\n", hipGetErrorString(_e), __FILE__, __LINE__); \
    return; } } while (0)

namespace {

typedef unsigned u32;
typedef unsigned long long u64;
typedef unsigned u32x2 __attribute__((ext_vector_type(2)));
typedef unsigned u32x4 __attribute__((ext_vector_type(4)));
typedef int i32x4 __attribute__((ext_vector_type(4)));
typedef float f32x2 __attribute__((ext_vector_type(2)));
typedef float f32x4 __attribute__((ext_vector_type(4)));
#define LAS __attribute__((address_space(3)))
#define GAS __attribute__((address_space(1)))

constexpr int NR = 8;
constexpr int MM = 4096, KK = 4096, NN = 16384;
constexpr int KS = KK / NR;
constexpr int NC = NN / NR;
constexpr int NWG = 256, NTHR = 512;

constexpr size_t OFF_CNT = 0;
constexpr size_t OFF_TMO = 32768;
constexpr size_t OFF_CLAIM = 36864;
constexpr size_t OFF_GATE = 40960;
constexpr size_t OFF_CLAIM2 = 49152;
constexpr size_t OFF_GATE2 = 53248;
constexpr size_t OFF_INFLAG = 524288;
constexpr size_t OFF_OFLAG = 131072;
constexpr size_t OFF_DBG2 = 262144;
constexpr size_t OFF_DBG = 1u << 20;
constexpr size_t OFF_XS = 2u << 20;
constexpr size_t XS_BYTES = 16u << 20;
constexpr size_t OFF_WS = OFF_XS + 2 * XS_BYTES;
constexpr size_t WS_BYTES = 8u << 20;
constexpr size_t OFF_OSLOT = OFF_WS + 2 * WS_BYTES;
constexpr u32 OSLOT = 256 + 8192;
constexpr size_t WS_NEED = OFF_OSLOT + (size_t)NR * 16 * 8 * 4 * OSLOT;

constexpr size_t SLOTB = 131072;
constexpr int HTB = 16384;
constexpr int RSLOT = 24576, RING = 6;
constexpr int LDS_SNAP = RING * RSLOT;
constexpr int LDS_BYTES = LDS_SNAP + 1024;
constexpr u64 DEADLINE = 30000000ull;

#define SCH_E "w0a0w1a1w2a2w3a3w4a4w5a5b0w6a6b1w7a7b2b3b4b5b6b7"
#define SCH_O SCH_E
struct Sched { u64 qof[3]; u64 typ; u64 rnd0, rnd1; };
constexpr Sched make_sched(const char* p) {
    Sched s{{0, 0, 0}, 0, 0, 0};
    for (int q = 0; q < 24; ++q) {
        const int t = p[2 * q] == 'w' ? 0 : p[2 * q] == 'a' ? 1 : 2, r = p[2 * q + 1] - '0';
        s.qof[t] |= (u64)q << (8 * r);
        s.typ |= (u64)t << (2 * q);
        if (q < 16) s.rnd0 |= (u64)r << (4 * q); else s.rnd1 |= (u64)r << (4 * (q - 16));
    }
    return s;
}
constexpr Sched SC0 = make_sched(SCH_E), SC1 = make_sched(SCH_O);
constexpr u64 make_order() {
    int key[16] = {};
    for (int p = 0; p < 16; ++p) {
        const Sched& s = (p & 1) ? SC1 : SC0;
        const int qw = (int)((s.qof[0] >> (8 * (p >> 1))) & 255), qx = (int)((s.qof[1] >> (8 * (p >> 1))) & 255);
        key[p] = (qw > qx ? qw : qx) * 16 + p;
    }
    u64 o = 0;
    for (int p = 0; p < 16; ++p) {
        int pos = 0;
        for (int k = 0; k < 16; ++k) pos += key[k] < key[p];
        o |= (u64)p << (4 * pos);
    }
    return o;
}
constexpr u64 ORD = make_order();
#define NPW 6
#define GATE 725
#define G2SEG 2
#define G2N 48
#define BSPLIT 10
#define ASPLIT 0x01
#define OUTST 0
#define ESLEEP 32
#define XSLEEP 8
#define GSLEEP 8
constexpr u32 NUNIT = 768;
constexpr u32 NPL = 8 * NPW;

struct Params {
    const char* x; const char* w; const float* sx; const float* sw;
    float* out;
    char* ws[NR];
    int rank;
};

__device__ __forceinline__ void st16_sys(void* p, u32x4 v) {
    asm volatile("global_store_dwordx4 %0, %1, off sc0 sc1\n\ts_nop 1" :: "v"(p), "v"(v) : "memory");
}
#define WAIT_V(n) asm volatile("s_waitcnt vmcnt(" #n ")" ::: "memory")
#define WAIT_L(n) asm volatile("s_waitcnt lgkmcnt(" #n ")" ::: "memory")
#define BAR __builtin_amdgcn_s_barrier()
#define SCHED __builtin_amdgcn_sched_barrier(0)

__device__ __forceinline__ u32 ld_sys32(const char* p) {
    return __hip_atomic_load((const u32*)p, __ATOMIC_RELAXED, __HIP_MEMORY_SCOPE_SYSTEM);
}
__device__ __forceinline__ bool timed_out(char* ws, u32 call) {
    return __hip_atomic_load((const u32*)(ws + OFF_TMO), __ATOMIC_RELAXED, __HIP_MEMORY_SCOPE_AGENT) == call;
}
__device__ __forceinline__ bool give_up(char* ws, u32 call, u64 t0, u32& pass) {
    if ((++pass & 255u) == 0 && timed_out(ws, call)) return true;
    if (__builtin_amdgcn_s_memrealtime() - t0 > DEADLINE) {
        __hip_atomic_store((u32*)(ws + OFF_TMO), call, __ATOMIC_RELAXED, __HIP_MEMORY_SCOPE_AGENT);
        return true;
    }
    return false;
}
template <int SLEEP> __device__ __forceinline__ void wait_words(char* ws, const char* p, u32 call) {
    const u64 t0 = __builtin_amdgcn_s_memrealtime();
    u32 pass = 0;
    while (__builtin_amdgcn_ballot_w64(ld_sys32(p) < call) != 0) {
        if (give_up(ws, call, t0, pass)) break;
        __builtin_amdgcn_s_sleep(SLEEP);
    }
}
__device__ __forceinline__ u32 wave_incl_scan(u32 v) {
    v += (u32)__builtin_amdgcn_update_dpp(0, (int)v, 0x111, 0xf, 0xf, false);
    v += (u32)__builtin_amdgcn_update_dpp(0, (int)v, 0x112, 0xf, 0xf, false);
    v += (u32)__builtin_amdgcn_update_dpp(0, (int)v, 0x114, 0xf, 0xf, false);
    v += (u32)__builtin_amdgcn_update_dpp(0, (int)v, 0x118, 0xf, 0xf, false);
    v += (u32)__builtin_amdgcn_update_dpp(0, (int)v, 0x142, 0xa, 0xf, false);
    v += (u32)__builtin_amdgcn_update_dpp(0, (int)v, 0x143, 0xc, 0xf, false);
    return v;
}

#define STAMP_PARAM
#define STAMP_ARG
#define STAMP_DECL
#define STAMP() do {} while (0)
#define STAMP_AT(k, val) do {} while (0)
#define STAMP_SET(k) do {} while (0)
#define STAMP_FLUSH(ws) do {} while (0)

__device__ __forceinline__ int unit_typ(u32 n, int cl) { return (int)(((cl ? SC1.typ : SC0.typ) >> (2 * (n >> 5))) & 3); }
__device__ __forceinline__ int unit_rnd(u32 n, int cl) {
    return (int)(((n < 512 ? (cl ? SC1.rnd0 : SC0.rnd0) : (cl ? SC1.rnd1 : SC0.rnd1)) >> (4 * ((n >> 5) & 15))) & 15);
}
__device__ __forceinline__ const char* unit_src(const Params& P, u32 n, int dst, int cl) {
    const int u = (int)(n >> 2) & 7, wv = (int)n & 3, typ = unit_typ(n, cl), rnd = unit_rnd(n, cl);
    return typ == 0 ? P.w + (size_t)(rnd * 64) * NN + dst * NC + u * 256 + wv * 64
                    : P.x + (size_t)((2 * u + typ - 1) * 256 + wv * 64) * KS + rnd * 64;
}
#define UNIT_LOADS \
    "s_cmp_lg_u32 %[typ], 0\n\ts_cbranch_scc1 1f\n\t" \
    "global_load_dword v8, %[vw], %[sb]\n\tv_add_u32 %[tmp], 0x4000, %[vw]\n\t" \
    "global_load_dword v9, %[tmp], %[sb]\n\tv_add_u32 %[tmp], 0x4000, %[tmp]\n\t" \
    "global_load_dword v10, %[tmp], %[sb]\n\tv_add_u32 %[tmp], 0x4000, %[tmp]\n\t" \
    "global_load_dword v11, %[tmp], %[sb]\n\tv_add_u32 %[tmp], 0x4000, %[tmp]\n\t" \
    "global_load_dword v12, %[tmp], %[sb]\n\tv_add_u32 %[tmp], 0x4000, %[tmp]\n\t" \
    "global_load_dword v13, %[tmp], %[sb]\n\tv_add_u32 %[tmp], 0x4000, %[tmp]\n\t" \
    "global_load_dword v14, %[tmp], %[sb]\n\tv_add_u32 %[tmp], 0x4000, %[tmp]\n\t" \
    "global_load_dword v15, %[tmp], %[sb]\n\tv_add_u32 %[tmp], 0x4000, %[tmp]\n\t" \
    "global_load_dword v16, %[tmp], %[sb]\n\tv_add_u32 %[tmp], 0x4000, %[tmp]\n\t" \
    "global_load_dword v17, %[tmp], %[sb]\n\tv_add_u32 %[tmp], 0x4000, %[tmp]\n\t" \
    "global_load_dword v18, %[tmp], %[sb]\n\tv_add_u32 %[tmp], 0x4000, %[tmp]\n\t" \
    "global_load_dword v19, %[tmp], %[sb]\n\tv_add_u32 %[tmp], 0x4000, %[tmp]\n\t" \
    "global_load_dword v20, %[tmp], %[sb]\n\tv_add_u32 %[tmp], 0x4000, %[tmp]\n\t" \
    "global_load_dword v21, %[tmp], %[sb]\n\tv_add_u32 %[tmp], 0x4000, %[tmp]\n\t" \
    "global_load_dword v22, %[tmp], %[sb]\n\tv_add_u32 %[tmp], 0x4000, %[tmp]\n\t" \
    "global_load_dword v23, %[tmp], %[sb]\n\ts_branch 2f\n" \
    "1:\n\ts_cmp_eq_u32 %[typ], 3\n\ts_cbranch_scc1 2f\n\t" \
    "global_load_dwordx4 v[8:11], %[vx], %[sb]\n\tv_add_u32 %[tmp], 0x2000, %[vx]\n\t" \
    "global_load_dwordx4 v[12:15], %[tmp], %[sb]\n\tv_add_u32 %[tmp], 0x2000, %[tmp]\n\t" \
    "global_load_dwordx4 v[16:19], %[tmp], %[sb]\n\tv_add_u32 %[tmp], 0x2000, %[tmp]\n\t" \
    "global_load_dwordx4 v[20:23], %[tmp], %[sb]\n" \
    "2:\n\t"

__device__ __forceinline__ u32 pusher(const Params& P, int dst, int widx, int lane, u32* callcnt STAMP_PARAM) {
    char* wsd = P.ws[dst];
    const int rel = (P.rank - dst) & 7, cl = rel >> 2;
    u32* cnt = (u32*)(P.ws[P.rank] + OFF_CLAIM + (size_t)dst * 128);
    u32* flags = (u32*)(wsd + OFF_INFLAG + (size_t)P.rank * (24 * 8 * 128));
    u32* gate = (u32*)(P.ws[P.rank] + OFF_GATE + (size_t)(dst * 8 + (lane & 7)) * 128);
    const u32 vw = (u32)((lane >> 4) * 16 * NN + (lane & 15) * 4), vx = (u32)((lane >> 2) * KS + (((lane & 3) ^ ((lane >> 5) << 1)) << 4));
    u32 n = (u32)widx, n1 = (u32)widx + NPL;
    u32x4 L0, L1, L2, L3;
    u32 t, tmp;
#define FETCH(counter, nn) asm volatile("s_mov_b64 exec, 1\n\tglobal_atomic_add %[t], %[c], %[one], off sc0\n\ts_mov_b64 exec, -1\n\t" \
        UNIT_LOADS "s_waitcnt vmcnt(0)" \
        : "=&{v[8:11]}"(L0), "=&{v[12:15]}"(L1), "=&{v[16:19]}"(L2), "=&{v[20:23]}"(L3), [t] "=&v"(t), [tmp] "=&v"(tmp) \
        : [c] "v"(counter), [one] "v"(1u), [typ] "s"((nn) < NUNIT ? unit_typ(nn, cl) : 3), [sb] "s"(unit_src(P, (nn) < NUNIT ? (nn) : 0u, dst, cl)), \
          [vw] "v"(vw), [vx] "v"(vx) : "memory", "scc")
    FETCH(callcnt, n);
    const u32 call = (u32)__builtin_amdgcn_readfirstlane((int)t) + 1;
    STAMP();
    const u32 par = call & 1;
    const u32 cbase = (call - 1) * NUNIT - 2 * NPL;
#pragma unroll 1
    while (n < NUNIT) {
        const int u = (int)(n >> 2) & 7, wv = (int)n & 3, typ = unit_typ(n, cl), rnd = unit_rnd(n, cl);
        u32x4 S[4] = {L0, L1, L2, L3};
        char* d0; int dstep2;
        if (typ == 0) {
            const u32x4 T[4] = {L0, L1, L2, L3};
#pragma unroll
            for (int k = 0; k < 4; ++k) {
                const u32 a = T[k].x, b = T[k].y, c = T[k].z, d = T[k].w;
                const u32 t0 = __builtin_amdgcn_perm(b, a, 0x05010400u), t1 = __builtin_amdgcn_perm(b, a, 0x07030602u);
                const u32 t2 = __builtin_amdgcn_perm(d, c, 0x05010400u), t3 = __builtin_amdgcn_perm(d, c, 0x07030602u);
                S[0][k] = __builtin_amdgcn_perm(t2, t0, 0x05040100u); S[1][k] = __builtin_amdgcn_perm(t2, t0, 0x07060302u);
                S[2][k] = __builtin_amdgcn_perm(t3, t1, 0x05040100u); S[3][k] = __builtin_amdgcn_perm(t3, t1, 0x07060302u);
            }
            d0 = wsd + OFF_WS + par * WS_BYTES + (size_t)((rnd * 8 + rel) * 8 + u) * 16384 + wv * 2048;
            dstep2 = 8192;
        } else {
            d0 = wsd + OFF_XS + par * XS_BYTES + (size_t)(((typ - 1) * 64 + rnd * 8 + rel) * 8 + u) * 16384 + wv * 4096;
            dstep2 = 2048;
        }
        d0 += lane * 16;
        st16_sys(d0, S[0]); st16_sys(d0 + 1024, S[1]); st16_sys(d0 + dstep2, S[2]); st16_sys(d0 + dstep2 + 1024, S[3]);
        FETCH(cnt, n1);
        if (lane == 0) __hip_atomic_store(flags + (n >> 2) * 32 + (n & 3), call, __ATOMIC_RELAXED, __HIP_MEMORY_SCOPE_SYSTEM);
        STAMP();
        n = n1;
        n1 = (u32)__builtin_amdgcn_readfirstlane((int)t) - cbase;
        if (n1 == GATE - 1 + 2 * NPL && lane < 8) __hip_atomic_store(gate, call, __ATOMIC_RELAXED, __HIP_MEMORY_SCOPE_SYSTEM);
    }
#undef FETCH
    return call;
}

__device__ __forceinline__ void expander(const Params& P, LAS char* lds, int src, int cb, int first, int last, int wid, int lane, u32 call STAMP_PARAM) {
    char* ws = P.ws[P.rank];
    const float sc = __builtin_fabsf(P.sx[0] * P.sw[0]);
    LAS char* scr = lds + wid * 16384;
    const int wr = wid >> 2, wc = wid & 3, fr = lane & 15, fq = lane >> 4;
    float* orow = P.out + (size_t)(wr * 64 + fq * 4) * NN + src * NC + cb * 256 + wc * 64 + fr * 4;
#pragma unroll 1
    for (int it8 = first; it8 <= last; ++it8) {
        const int h = it8 >> 2, sg = it8 & 3;
        const int wt = (src * 16 + h * 8 + cb) * 8 + wid;
        const u64* flag = (const u64*)(ws + OFF_OFLAG + (size_t)wt * 128);
        u64 fv;
        {
            const u64 t0 = __builtin_amdgcn_s_memrealtime();
            u32 pass = 0;
            for (;;) {
                fv = __hip_atomic_load(flag + sg, __ATOMIC_RELAXED, __HIP_MEMORY_SCOPE_SYSTEM);
                if ((u32)__builtin_amdgcn_readfirstlane((int)(fv >> 32)) >= call) break;
                if (give_up(ws, call, t0, pass)) break;
                __builtin_amdgcn_s_sleep(XSLEEP);
            }
        }
        STAMP();
        const char* slot = ws + OFF_OSLOT + (size_t)(wt * 4 + sg) * OSLOT;
        const char* pc = slot + 256 + lane * 16;
        u32 mask, tmo; u32x4 c0, c1, c2;
        asm volatile("global_load_dword %0, %5, off sc0 sc1\n\t"
                     "global_load_dwordx4 %1, %6, off sc0 sc1\n\t"
                     "global_load_dwordx4 %2, %6, off offset:1024 sc0 sc1\n\t"
                     "global_load_dwordx4 %3, %6, off offset:2048 sc0 sc1\n\t"
                     "global_load_dword %4, %7, off sc1\n\t"
                     "s_waitcnt vmcnt(0)"
                     : "=&v"(mask), "=&v"(c0), "=&v"(c1), "=&v"(c2), "=&v"(tmo) : "v"(slot + lane * 4), "v"(pc), "v"(ws + OFF_TMO) : "memory");
        u32 info = (u32)__builtin_amdgcn_readfirstlane((int)(u32)fv);
        if ((u32)__builtin_amdgcn_readfirstlane((int)tmo) == call) info = 2u << 16;
        const u32 mode = info >> 16;
        u32 n = info & 0xffffu; if (n > 2048u) n = 2048u;
        float* o = orow + (size_t)(h * 256 + (sg >> 1) * 128 + (sg & 1) * 32) * NN;
        if (mode >= 2) {
            const float qn = __builtin_nanf("");
#pragma unroll
            for (int g = 0; g < 8; ++g) *(f32x4*)(o + (size_t)((g >> 2) * 16 + (g & 3)) * NN) = (f32x4){qn, qn, qn, qn};
            continue;
        }
        if (mode == 0) {
            auto unpack = [&](u32x4 c, int q) {
                const u32 m21 = 0x1fffffu;
                LAS f32x2* d = (LAS f32x2*)(scr + q * 24);
                d[0] = (f32x2){(float)(c.x & m21) * sc, (float)(__builtin_amdgcn_alignbit(c.y, c.x, 21) & m21) * sc};
                d[1] = (f32x2){(float)((c.y >> 10) & m21) * sc, (float)(c.z & m21) * sc};
                d[2] = (f32x2){(float)(__builtin_amdgcn_alignbit(c.w, c.z, 21) & m21) * sc, (float)((c.w >> 10) & m21) * sc};
            };
            unpack(c0, lane); unpack(c1, 64 + lane); unpack(c2, 128 + lane);
#pragma unroll 1
            for (u32 it = 3; it * 384u < n; ++it) {
                u32x4 c;
                if (it * 64 + lane < 342u) {
                    asm volatile("global_load_dwordx4 %0, %1, off sc0 sc1\n\ts_waitcnt vmcnt(0)" : "=&v"(c) : "v"(pc + it * 1024) : "memory");
                    unpack(c, it * 64 + lane);
                }
            }
        } else {
            auto unraw = [&](u32x4 c, int q) {
                *(LAS f32x4*)(scr + q * 16) = (f32x4){(float)c.x * sc, (float)c.y * sc, (float)c.z * sc, (float)c.w * sc};
            };
            unraw(c0, lane); unraw(c1, 64 + lane); unraw(c2, 128 + lane);
#pragma unroll 1
            for (u32 it = 3; it * 256u < n; ++it) {
                u32x4 c;
                asm volatile("global_load_dwordx4 %0, %1, off sc0 sc1\n\ts_waitcnt vmcnt(0)" : "=&v"(c) : "v"(pc + it * 1024) : "memory");
                unraw(c, it * 64 + lane);
            }
        }
        u32 incl = wave_incl_scan((u32)__builtin_popcount(mask));
        if (incl > 2048u) incl = 2048u;
        u32 a = incl * 4;
        __builtin_amdgcn_wave_barrier();
        u32 v[32];
#pragma unroll
        for (int e = 31; e >= 0; --e) {
            const int t = __builtin_amdgcn_sbfe((int)mask, e, 1);
            a += (u32)t << 2;
            v[e] = *(LAS u32*)(scr + (a & 0x3fffu)) & (u32)t;
        }
#pragma unroll
        for (int g = 0; g < 8; ++g) {
            const u32x4 ov = (u32x4){v[4 * g], v[4 * g + 1], v[4 * g + 2], v[4 * g + 3]};
            float* op = o + (size_t)((g >> 2) * 16 + (g & 3)) * NN;
            *(u32x4*)op = ov;
        }
        __builtin_amdgcn_wave_barrier();
    }
}

__device__ __forceinline__ u32 pack_seg(const int (&v)[32], LAS char* scr, int lane) {
    u32 mask = 0; int mx = 0;
#pragma unroll
    for (int e = 0; e < 32; ++e) { mask |= (v[e] > 0 ? 1u : 0u) << e; mx = v[e] > mx ? v[e] : mx; }
    const u32 cnt = (u32)__builtin_popcount(mask);
    const u32 incl = wave_incl_scan(cnt);
    const u32 n = (u32)__builtin_amdgcn_readlane((int)incl, 63);
    const bool raw = __builtin_amdgcn_ballot_w64(mx >= (1 << 21)) != 0;
    u32 a = (incl - cnt) * 4;
    __builtin_amdgcn_wave_barrier();
#pragma unroll
    for (int e = 0; e < 32; ++e) if (v[e] > 0) { *(LAS int*)(scr + a) = v[e]; a += 4; }
    *(LAS u32*)(scr + 12288 + lane * 4) = mask;
    __builtin_amdgcn_wave_barrier();
    return n | (raw ? 1u << 16 : 0u);
}
__device__ __forceinline__ void send_seg(const u32 SPLIT, char* slot, LAS char* scr, u32 info, int lane) {
    const u32 n = info & 0xffffu;
    if (lane < 16) st16_sys(slot + lane * 16, *(const LAS u32x4*)(scr + 12288 + lane * 16));
    if (info < 0x10000u) {
        const u32 nch = (n + 5) / 6;
#pragma unroll 1
        for (u32 q = lane; q < nch; q += 64) {
            const LAS u32x2* s = (const LAS u32x2*)(scr + q * 24);
            const u32x2 p0 = s[0], p1 = s[1], p2 = s[2];
            u32x4 c;
            c.x = p0.x | (p0.y << 21); c.y = (p0.y >> 11) | (p1.x << 10);
            c.z = p1.y | (p2.x << 21); c.w = (p2.x >> 11) | (p2.y << 10);
            st16_sys(slot + 256 + q * 16, c);
            if (q < 64 * SPLIT) WAIT_V(0);
        }
    } else {
        const u32 nch = (n + 3) / 4;
#pragma unroll 1
        for (u32 q = lane; q < nch; q += 64) st16_sys(slot + 256 + q * 16, *(const LAS u32x4*)(scr + q * 16));
    }
}
__device__ __forceinline__ void publish(u64* flag, u32 call, u32 info, int lane) {
    if (lane == 0) __hip_atomic_store(flag, ((u64)call << 32) | info, __ATOMIC_RELAXED, __HIP_MEMORY_SCOPE_SYSTEM);
}
__device__ __forceinline__ void pass_gate(char* ws, size_t off, bool remote, int wid, u32 call) {
    if (wid == 0 && remote) wait_words<GSLEEP>(ws, ws + off, call);
    asm volatile("" ::: "memory");
    BAR;
    asm volatile("" ::: "memory");
}

__device__ __forceinline__ const char* pair_flag_addr(const char* ws, int rank, int Pn, int lane, int cb, int own) {
    const int s = (rank + (Pn & 1) * 4 + ((lane >> 3) & 3)) & 7, i = Pn >> 1;
    const u64 tw = (Pn & 1) ? SC1.qof[0] : SC0.qof[0], tx = (Pn & 1) ? SC1.qof[1] : SC0.qof[1];
    const int qw = (int)((tw >> (8 * i)) & 255), qx = (int)((tx >> (8 * i)) & 255);
    return ws + OFF_INFLAG + ((s * 24 + ((lane & 4) ? qx : qw)) * 8 + ((lane & 4) ? own : cb)) * 128 + (lane & 3) * 4;
}

__device__ __forceinline__ void gemm_full(const Params& P, LAS char* lds, int own, int cb, int wid, int lane, u32 call STAMP_PARAM) {
    char* ws = P.ws[P.rank];
    const int tid = threadIdx.x, wr = wid >> 2, wc = wid & 3, fr = lane & 15, fq = lane >> 4;
    const u32 par = call & 1;
    const char* cA = ws + OFF_XS + par * XS_BYTES + (size_t)own * 16384;
    const char* cB = ws + OFF_WS + par * WS_BYTES + (size_t)cb * 16384;
    const u32 voff = (u32)tid * 16;
    const u32 ldsw = (u32)wid * 1024;
    const int aoff = wr * 4096 + fr * 64 + ((fq ^ ((fr >> 3) << 1)) << 4), boff = wc * 2048 + lane * 16;
    STAMP();
#define SA(b, hh) (((b) * 2 + (hh)) * HTB)
#define SB(b, hh) ((4 + (b) * 2 + (hh)) * HTB)
#define STAGE(bufoff, gbase) do { _Pragma("unroll") for (int _i = 0; _i < 2; ++_i) \
        __builtin_amdgcn_global_load_lds((const GAS u32*)((gbase) + _i * SLOTB + voff), (LAS u32*)(lds + (bufoff) + ldsw + _i * 8192), 16, 0, 17); } while (0)
#define PIDX(Pn) ((int)(ORD >> (4 * (Pn))) & 15)
#define SNAP(Pn) do { if (wid == 0) __builtin_amdgcn_global_load_lds((const GAS u32*)pair_flag_addr(ws, P.rank, PIDX(Pn), lane, cb, own), \
        (LAS u32*)(lds + LDS_SNAP + ((Pn) & 1) * 256), 4, 0, 17); } while (0)
#define LDA(dst, b, hh) do { _Pragma("unroll") for (int m = 0; m < 4; ++m) _Pragma("unroll") for (int k = 0; k < 2; ++k) \
        dst[m][k] = *(const LAS i32x4*)(lds + SA(b, hh) + aoff + m * 1024 + k * 8192); } while (0)
#define LDB(dst, b, hh) do { _Pragma("unroll") for (int n = 0; n < 2; ++n) _Pragma("unroll") for (int k = 0; k < 2; ++k) \
        dst[n][k] = *(const LAS i32x4*)(lds + SB(b, hh) + boff + n * 1024 + k * 8192); } while (0)
#define MMA(ai, bj, At, Bt) do { __builtin_amdgcn_s_setprio(1); _Pragma("unroll") for (int m = 0; m < 4; ++m) _Pragma("unroll") for (int n = 0; n < 2; ++n) \
        _Pragma("unroll") for (int k = 0; k < 2; ++k) acc[ai][bj][m][n] = __builtin_amdgcn_mfma_i32_16x16x64_i8(At[m][k], Bt[n][k], acc[ai][bj][m][n], 0, 0, 0); \
        __builtin_amdgcn_s_setprio(0); } while (0)
    i32x4 acc[2][2][4][2];
#pragma unroll
    for (int a = 0; a < 2; ++a)
#pragma unroll
        for (int b = 0; b < 2; ++b)
#pragma unroll
            for (int m = 0; m < 4; ++m)
#pragma unroll
                for (int n = 0; n < 2; ++n) acc[a][b][m][n] = (i32x4){0, 0, 0, 0};
    i32x4 At[4][2], B0[2][2], B1[2][2];
    int Pc = 0;
    bool pf = false;
#pragma unroll 1
    for (;;) {
        const char* a0 = cA + (size_t)PIDX(Pc) * (4 * SLOTB); const char* b0 = cB + (size_t)PIDX(Pc) * (4 * SLOTB);
        if (!pf) {
            if (wid == 0) wait_words<GSLEEP>(ws, pair_flag_addr(ws, P.rank, PIDX(Pc), lane, cb, own), call);
            asm volatile("" ::: "memory");
            if (wr == 0 || Pc == 0) BAR;
            asm volatile("" ::: "memory");
            STAMP();
            if (Pc < 15) SNAP(Pc + 1);
            STAGE(SB(0, 0), b0); STAGE(SA(0, 0), a0); STAGE(SB(0, 1), b0 + 8192); STAGE(SA(0, 1), a0 + 8192);
            if (wr == 1) BAR;
            WAIT_V(4); BAR;
            STAGE(SB(1, 0), b0 + 2 * SLOTB); STAGE(SA(1, 0), a0 + 2 * SLOTB); STAGE(SB(1, 1), b0 + 2 * SLOTB + 8192);
            WAIT_V(6); BAR;
        }
        STAMP();
        pf = false;
        if (Pc < 15) {
            const u32 sv = *(const LAS u32*)(lds + LDS_SNAP + ((Pc + 1) & 1) * 256 + lane * 4);
            pf = __builtin_amdgcn_ballot_w64(sv < call) == 0;
        }
        const char* a2 = cA + (size_t)PIDX((Pc + 1) & 15) * (4 * SLOTB); const char* b2 = cB + (size_t)PIDX((Pc + 1) & 15) * (4 * SLOTB);
        LDB(B0, 0, 0); SCHED; LDA(At, 0, 0); STAGE(SA(1, 1), a0 + 2 * SLOTB + 8192);
        WAIT_L(8); BAR; WAIT_L(0); MMA(0, 0, At, B0); BAR; SCHED;
        LDB(B1, 0, 1); if (pf) STAGE(SB(0, 0), b2);
        BAR; WAIT_L(0); MMA(0, 1, At, B1); BAR;
        LDA(At, 0, 1); if (pf) STAGE(SA(0, 0), a2);
        BAR; WAIT_L(0); MMA(1, 0, At, B0); BAR; SCHED;
        if (pf) { STAGE(SB(0, 1), b2 + 8192); WAIT_V(6); } else { WAIT_V(0); }
        BAR; MMA(1, 1, At, B1); BAR;
        LDB(B0, 1, 0); SCHED; LDA(At, 1, 0);
        if (pf) { if (Pc < 14) SNAP(Pc + 2); STAGE(SA(0, 1), a2 + 8192); }
        WAIT_L(8); BAR; WAIT_L(0); MMA(0, 0, At, B0); BAR; SCHED;
        LDB(B1, 1, 1); if (pf) STAGE(SB(1, 0), b2 + 2 * SLOTB);
        BAR; WAIT_L(0); MMA(0, 1, At, B1); BAR;
        LDA(At, 1, 1); if (pf) STAGE(SA(1, 0), a2 + 2 * SLOTB);
        BAR; WAIT_L(0); MMA(1, 0, At, B0); BAR; SCHED;
        if (pf) { STAGE(SB(1, 1), b2 + 2 * SLOTB + 8192); WAIT_V(6); }
        BAR; MMA(1, 1, At, B1); BAR;
        if (++Pc == 16) break;
    }
    if (wr == 0) BAR;
    STAMP_SET(40);
    STAMP();

    const bool dead = timed_out(ws, call);
    if (P.sx[0] * P.sw[0] < 0.f) {
#pragma unroll
        for (int a = 0; a < 2; ++a)
#pragma unroll
            for (int b = 0; b < 2; ++b)
#pragma unroll
                for (int m = 0; m < 4; ++m)
#pragma unroll
                    for (int n = 0; n < 2; ++n) acc[a][b][m][n] = -acc[a][b][m][n];
    }
    char* wso = P.ws[own];
    const bool remote = own != P.rank;
    const int wt = (P.rank * 16 + cb) * 8 + wid;
    u64* oflag = (u64*)(wso + OFF_OFLAG + (size_t)wt * 128);
    LAS char* scr = lds + wid * 16384;
    u32 info_prev = 0;
#pragma unroll
    for (int sg = 0; sg < 4; ++sg) {
        int v[32];
#pragma unroll
        for (int e = 0; e < 32; ++e) v[e] = acc[sg >> 1][(e >> 1) & 1][(sg & 1) * 2 + (e >> 4)][e & 1][(e >> 2) & 3];
        const u32 info = pack_seg(v, scr, lane);
        if (sg == 0) {
            pass_gate(ws, OFF_GATE + (size_t)(own * 8 + cb) * 128, remote, wid, call);
            STAMP();
        } else {
            if (sg == G2SEG) {
                u32 t;
                asm volatile("s_mov_b64 exec, 1\n\tglobal_atomic_add %0, %1, %2, off sc0\n\ts_mov_b64 exec, -1\n\ts_waitcnt vmcnt(0)"
                             : "=&v"(t) : "v"(ws + OFF_CLAIM2 + (size_t)own * 128), "v"(1u) : "memory");
                if ((u32)__builtin_amdgcn_readfirstlane((int)t) == (call - 1) * 64 + (G2N - 1) && lane < 8)
                    __hip_atomic_store((u32*)(ws + OFF_GATE2 + (size_t)(own * 8 + lane) * 128), call, __ATOMIC_RELAXED, __HIP_MEMORY_SCOPE_SYSTEM);
            } else {
                WAIT_V(0);
            }
            publish(oflag + (sg - 1), call, info_prev, lane);
        }
        send_seg((ASPLIT >> (2 * sg)) & 3, wso + OFF_OSLOT + (size_t)(wt * 4 + sg) * OSLOT, scr, info, lane);
        info_prev = dead ? (2u << 16) : info;
        STAMP();
    }
    WAIT_V(0);
    publish(oflag + 3, call, info_prev, lane);
    STAMP();
#undef SA
#undef SB
#undef STAGE
#undef SNAP
#undef PIDX
#undef LDA
#undef LDB
#undef MMA
}

__device__ __forceinline__ const char* round_flag_addr(const char* ws, int rank, int r, int lane, int cb, int own, int hb) {
    const int s = (rank + (lane >> 3)) & 7;
    const int qw0 = (int)((SC0.qof[0] >> (8 * r)) & 255), qw1 = (int)((SC1.qof[0] >> (8 * r)) & 255);
    const int qx0 = (int)((SC0.qof[2] >> (8 * r)) & 255), qx1 = (int)((SC1.qof[2] >> (8 * r)) & 255);
    const int q = (lane & 4) ? (lane < 32 ? qx0 : qx1) : (lane < 32 ? qw0 : qw1);
    return ws + OFF_INFLAG + ((s * 24 + q) * 8 + ((lane & 4) ? own : cb)) * 128 + ((lane & 4) ? (lane & 3) : 2 * hb + (lane & 1)) * 4;
}

__device__ __forceinline__ void gemm_half(const Params& P, LAS char* lds, int own, int hb, int cb, int wid, int lane, u32 call STAMP_PARAM) {
    char* ws = P.ws[P.rank];
    const int rg = wid & 3, cg = wid >> 2, fr = lane & 15, fq = lane >> 4;
    const u32 par = call & 1;
    const char* cA = ws + OFF_XS + par * XS_BYTES + 64 * SLOTB + (size_t)own * 16384;
    const char* cB = ws + OFF_WS + par * WS_BYTES + (size_t)cb * 16384;
    const u32 voffA = (u32)threadIdx.x * 16, voffB = (u32)(cg * 8192 + hb * 4096 + rg * 1024 + lane * 16);
    const u32 ldsw = (u32)wid * 1024;
    const u32 aoff = rg * 4096 + fr * 64 + ((fq ^ ((fr >> 3) << 1)) << 4), boff = 16384 + cg * 2048 + lane * 16;
    STAMP();
#define STAGE(pos, slot) do { \
        __builtin_amdgcn_global_load_lds((const GAS u32*)(cA + (size_t)(slot) * SLOTB + voffA), (LAS u32*)(lds + (pos) + ldsw), 16, 0, 17); \
        __builtin_amdgcn_global_load_lds((const GAS u32*)(cA + (size_t)(slot) * SLOTB + 8192 + voffA), (LAS u32*)(lds + (pos) + ldsw + 8192), 16, 0, 17); \
        __builtin_amdgcn_global_load_lds((const GAS u32*)(cB + (size_t)(slot) * SLOTB + voffB), (LAS u32*)(lds + (pos) + ldsw + 16384), 16, 0, 17); } while (0)
    i32x4 acc[4][4];
#pragma unroll
    for (int m = 0; m < 4; ++m)
#pragma unroll
        for (int n = 0; n < 4; ++n) acc[m][n] = (i32x4){0, 0, 0, 0};
    int r = 0;
    bool pf = false;
    u32 pos = 0, prev = 0;
#pragma unroll 1
    for (;;) {
        if (!pf) {
            if (wid == 0) wait_words<GSLEEP>(ws, round_flag_addr(ws, P.rank, r, lane, cb, own, hb), call);
            asm volatile("" ::: "memory");
            if (cg == 0 || r == 0) BAR;
            asm volatile("" ::: "memory");
            STAMP();
#pragma unroll
            for (int i = 0; i < 5; ++i) STAGE(i * RSLOT, r * 8 + i);
            pos = 0; prev = 5 * RSLOT;
            WAIT_V(12); BAR;
            if (cg == 1) BAR;
        }
        STAMP();
#pragma unroll
        for (int i = 0; i < 8; ++i) {
            if (i == 0 && r < 7 && wid == 0)
                __builtin_amdgcn_global_load_lds((const GAS u32*)round_flag_addr(ws, P.rank, r + 1, lane, cb, own, hb), (LAS u32*)(lds + LDS_SNAP), 4, 0, 17);
            if (i == 3) {
                const u32 sv = *(const LAS u32*)(lds + LDS_SNAP + lane * 4);
                pf = r < 7 && __builtin_amdgcn_ballot_w64(sv < call) == 0;
            }
            i32x4 At[4], Bt[4];
#pragma unroll
            for (int n = 0; n < 4; ++n) Bt[n] = *(const LAS i32x4*)(lds + pos + boff + (n >> 1) * 4096 + (n & 1) * 1024);
#pragma unroll
            for (int m = 0; m < 4; ++m) At[m] = *(const LAS i32x4*)(lds + pos + aoff + m * 1024);
            SCHED;
            if (i < 3 || pf) STAGE(prev, r * 8 + i + 5);
            WAIT_L(0);
            if (i == 2) WAIT_V(9);
            else if (i < 3 || pf) WAIT_V(12);
            else if (i == 3) WAIT_V(9);
            else if (i == 4) WAIT_V(6);
            else if (i == 5) WAIT_V(3);
            else WAIT_V(0);
            BAR; SCHED;
            __builtin_amdgcn_s_setprio(1);
#pragma unroll
            for (int m = 0; m < 4; ++m)
#pragma unroll
                for (int n = 0; n < 4; ++n) acc[m][n] = __builtin_amdgcn_mfma_i32_16x16x64_i8(At[m], Bt[n], acc[m][n], 0, 0, 0);
            __builtin_amdgcn_s_setprio(0);
            SCHED; BAR;
            prev = pos; pos = pos == (RING - 1) * RSLOT ? 0 : pos + RSLOT;
        }
        if (++r == 8) break;
    }
#undef STAGE
    if (cg == 0) BAR;
    STAMP_SET(40);
    STAMP();

    const bool dead = timed_out(ws, call);
    if (P.sx[0] * P.sw[0] < 0.f) {
#pragma unroll
        for (int m = 0; m < 4; ++m)
#pragma unroll
            for (int n = 0; n < 4; ++n) acc[m][n] = -acc[m][n];
    }
    char* wso = P.ws[own];
    const int wt = (P.rank * 16 + 8 + cb) * 8 + (rg & 1) * 4 + hb * 2 + cg, sg0 = (rg >> 1) * 2;
    u64* oflag = (u64*)(wso + OFF_OFLAG + (size_t)wt * 128) + sg0;
    LAS char* scr = lds + wid * 16384;
    u32 info_prev = 0;
#pragma unroll
    for (int sg = 0; sg < 2; ++sg) {
        int v[32];
#pragma unroll
        for (int e = 0; e < 32; ++e) v[e] = acc[sg * 2 + (e >> 4)][e & 3][(e >> 2) & 3];
        const u32 info = pack_seg(v, scr, lane);
        if (sg == 0) {
            pass_gate(ws, OFF_GATE2 + (size_t)(own * 8 + cb) * 128, own != P.rank, wid, call);
            STAMP();
        } else {
            WAIT_V(0);
            publish(oflag, call, info_prev, lane);
        }
        send_seg((BSPLIT >> (2 * sg)) & 3, wso + OFF_OSLOT + (size_t)(wt * 4 + sg0 + sg) * OSLOT, scr, info, lane);
        info_prev = dead ? (2u << 16) : info;
        STAMP();
    }
    WAIT_V(0);
    publish(oflag + 1, call, info_prev, lane);
    STAMP_SET(46);
    STAMP();
    STAMP_FLUSH(ws);
}

__global__ __launch_bounds__(NTHR, 2) void fused_kernel(const Params P) {
    extern __shared__ __attribute__((aligned(16))) char smem[];
    LAS char* lds = (LAS char*)smem;
    const int wid = __builtin_amdgcn_readfirstlane((int)(threadIdx.x >> 6)), lane = threadIdx.x & 63;
    const int xcd = blockIdx.x & 7, j = blockIdx.x >> 3;
    char* ws = P.ws[P.rank];
    STAMP_DECL;
    STAMP();
    u32* callcnt = (u32*)(ws + OFF_CNT + (size_t)blockIdx.x * 128) + wid;
    u32 call = 0;
    if (j < 8 && wid < NPW) {
        call = pusher(P, (P.rank + j) & 7, xcd * NPW + wid, lane, callcnt STAMP_ARG);
    } else {
        if (lane == 0) call = __hip_atomic_fetch_add(callcnt, 1u, __ATOMIC_RELAXED, __HIP_MEMORY_SCOPE_AGENT) + 1;
        call = (u32)__builtin_amdgcn_readfirstlane((int)call);
        if (j >= 16) { gemm_half(P, lds, j & 7, (j >> 3) & 1, xcd, wid, lane, call STAMP_ARG); return; }
        if (j >= 8) gemm_full(P, lds, j & 7, xcd, wid, lane, call STAMP_ARG);
        else STAMP();
    }
    if (j < 8) {
        STAMP_AT(40, __builtin_amdgcn_s_memrealtime()); STAMP_SET(41);
        if (wid == 0) wait_words<ESLEEP>(ws, ws + OFF_GATE + (size_t)((j == P.rank ? (j + 1) & 7 : j) * 8 + xcd) * 128, call);
        asm volatile("" ::: "memory");
        BAR;
        asm volatile("" ::: "memory");
    }
    expander(P, lds, j & 7, xcd, j < 8 ? 0 : 6, j < 8 ? 5 : 7, wid, lane, call STAMP_ARG);
    STAMP_SET(49);
    STAMP();
    STAMP_FLUSH(ws);
}

}

extern "C" void kernel_launch(int rank, int nranks, ncclComm_t comm, hipStream_t stream,
                              void* const* d_in, const int* in_sizes, int n_in,
                              void* d_out, int out_size, const int* in_dtypes, int out_dtype,
                              void* const* d_out_all, void* const* d_ws_all, size_t ws_bytes) {
    (void)comm; (void)in_dtypes; (void)out_dtype; (void)d_out_all;
    if (nranks != NR || (unsigned)rank >= (unsigned)NR || n_in != 4 || ws_bytes < WS_NEED ||
        in_sizes[0] != MM * KS || in_sizes[1] != KS * NN || out_size != (MM / NR) * NN) {
        fprintf(stderr, "kernel_launch: unexpected problem shape\n");
        return;
    }
    static bool attr_set[64] = {};
    int dev = 0;
    HIP_CHECK(hipGetDevice(&dev));
    if (!attr_set[dev & 63]) {
        HIP_CHECK(hipFuncSetAttribute((const void*)fused_kernel, hipFuncAttributeMaxDynamicSharedMemorySize, LDS_BYTES));
        attr_set[dev & 63] = true;
    }
    Params P;
    P.x = (const char*)d_in[0]; P.w = (const char*)d_in[1];
    P.sx = (const float*)d_in[2]; P.sw = (const float*)d_in[3];
    P.out = (float*)d_out;
    for (int p = 0; p < NR; ++p) P.ws[p] = (char*)d_ws_all[p];
    P.rank = rank;
    hipLaunchKernelGGL(fused_kernel, dim3(NWG), dim3(NTHR), LDS_BYTES, stream, P);
    HIP_CHECK(hipGetLastError());
}
```

```cpp
#include <hip/hip_runtime.h>
#include <rccl/rccl.h>
#include <cstdio>
#include <cstdint>

#define HIP_CHECK(expr) do { hipError_t _e = (expr); if (_e != hipSuccess) { \
    fprintf(stderr, "HIP error %s at %s:%d\n", hipGetErrorString(_e), __FILE__, __LINE__); \
    return; } } while (0)

namespace {

typedef unsigned u32;
typedef unsigned long long u64;
typedef unsigned u32x2 __attribute__((ext_vector_type(2)));
typedef unsigned u32x4 __attribute__((ext_vector_type(4)));
typedef int i32x4 __attribute__((ext_vector_type(4)));
typedef float f32x2 __attribute__((ext_vector_type(2)));
typedef float f32x4 __attribute__((ext_vector_type(4)));
#define LAS __attribute__((address_space(3)))
#define GAS __attribute__((address_space(1)))

constexpr int NR = 8;
constexpr int MM = 4096, KK = 4096, NN = 16384;
constexpr int KS = KK / NR;
constexpr int NC = NN / NR;
constexpr int NWG = 256, NTHR = 512;

constexpr size_t OFF_CNT = 0;
constexpr size_t OFF_TMO = 32768;
constexpr size_t OFF_CLAIM = 36864;
constexpr size_t OFF_GATE = 40960;
constexpr size_t OFF_CLAIM2 = 49152;
constexpr size_t OFF_GATE2 = 53248;
constexpr size_t OFF_INFLAG = 524288;
constexpr size_t OFF_OFLAG = 131072;
constexpr size_t OFF_DBG2 = 262144;
constexpr size_t OFF_DBG = 1u << 20;
constexpr size_t OFF_XS = 2u << 20;
constexpr size_t XS_BYTES = 16u << 20;
constexpr size_t OFF_WS = OFF_XS + 2 * XS_BYTES;
constexpr size_t WS_BYTES = 8u << 20;
constexpr size_t OFF_OSLOT = OFF_WS + 2 * WS_BYTES;
constexpr u32 OSLOT = 256 + 8192;
constexpr size_t WS_NEED = OFF_OSLOT + (size_t)NR * 16 * 8 * 4 * OSLOT;

constexpr size_t SLOTB = 131072;
constexpr int HTB = 16384;
constexpr int RSLOT = 24576, RING = 6;
constexpr int LDS_SNAP = RING * RSLOT;
constexpr int LDS_BYTES = LDS_SNAP + 1024;
constexpr u64 DEADLINE = 30000000ull;

#define SCH_E "w0a0w1a1w2a2w3a3w4a4w5a5b0w6a6b1w7a7b2b3b4b5b6b7"
#define SCH_O SCH_E
struct Sched { u64 qof[3]; u64 typ; u64 rnd0, rnd1; };
constexpr Sched make_sched(const char* p) {
    Sched s{{0, 0, 0}, 0, 0, 0};
    for (int q = 0; q < 24; ++q) {
        const int t = p[2 * q] == 'w' ? 0 : p[2 * q] == 'a' ? 1 : 2, r = p[2 * q + 1] - '0';
        s.qof[t] |= (u64)q << (8 * r);
        s.typ |= (u64)t << (2 * q);
        if (q < 16) s.rnd0 |= (u64)r << (4 * q); else s.rnd1 |= (u64)r << (4 * (q - 16));
    }
    return s;
}
constexpr Sched SC0 = make_sched(SCH_E), SC1 = make_sched(SCH_O);
constexpr u64 make_order() {
    int key[16] = {};
    for (int p = 0; p < 16; ++p) {
        const Sched& s = (p & 1) ? SC1 : SC0;
        const int qw = (int)((s.qof[0] >> (8 * (p >> 1))) & 255), qx = (int)((s.qof[1] >> (8 * (p >> 1))) & 255);
        key[p] = (qw > qx ? qw : qx) * 16 + p;
    }
    u64 o = 0;
    for (int p = 0; p < 16; ++p) {
        int pos = 0;
        for (int k = 0; k < 16; ++k) pos += key[k] < key[p];
        o |= (u64)p << (4 * pos);
    }
    return o;
}
constexpr u64 ORD = make_order();
#define NPW 3
#define GATE 725
#define G2SEG 2
#define G2N 48
#define BSPLIT 10
#define ASPLIT 0x01
#define OUTST 2
#define ESLEEP 32
#define XSLEEP 8
#define GSLEEP 8
constexpr u32 NUNIT = 768;
constexpr u32 NPL = 8 * NPW;

struct Params {
    const char* x; const char* w; const float* sx; const float* sw;
    float* out;
    char* ws[NR];
    int rank;
};

__device__ __forceinline__ void st16_sys(void* p, u32x4 v) {
    asm volatile("global_store_dwordx4 %0, %1, off sc0 sc1\n\ts_nop 1" :: "v"(p), "v"(v) : "memory");
}
#define WAIT_V(n) asm volatile("s_waitcnt vmcnt(" #n ")" ::: "memory")
#define WAIT_L(n) asm volatile("s_waitcnt lgkmcnt(" #n ")" ::: "memory")
#define BAR __builtin_amdgcn_s_barrier()
#define SCHED __builtin_amdgcn_sched_barrier(0)

__device__ __forceinline__ u32 ld_sys32(const char* p) {
    return __hip_atomic_load((const u32*)p, __ATOMIC_RELAXED, __HIP_MEMORY_SCOPE_SYSTEM);
}
__device__ __forceinline__ bool timed_out(char* ws, u32 call) {
    return __hip_atomic_load((const u32*)(ws + OFF_TMO), __ATOMIC_RELAXED, __HIP_MEMORY_SCOPE_AGENT) == call;
}
__device__ __forceinline__ bool give_up(char* ws, u32 call, u64 t0, u32& pass) {
    if ((++pass & 255u) == 0 && timed_out(ws, call)) return true;
    if (__builtin_amdgcn_s_memrealtime() - t0 > DEADLINE) {
        __hip_atomic_store((u32*)(ws + OFF_TMO), call, __ATOMIC_RELAXED, __HIP_MEMORY_SCOPE_AGENT);
        return true;
    }
    return false;
}
template <int SLEEP> __device__ __forceinline__ void wait_words(char* ws, const char* p, u32 call) {
    const u64 t0 = __builtin_amdgcn_s_memrealtime();
    u32 pass = 0;
    while (__builtin_amdgcn_ballot_w64(ld_sys32(p) < call) != 0) {
        if (give_up(ws, call, t0, pass)) break;
        __builtin_amdgcn_s_sleep(SLEEP);
    }
}
__device__ __forceinline__ u32 wave_incl_scan(u32 v) {
    v += (u32)__builtin_amdgcn_update_dpp(0, (int)v, 0x111, 0xf, 0xf, false);
    v += (u32)__builtin_amdgcn_update_dpp(0, (int)v, 0x112, 0xf, 0xf, false);
    v += (u32)__builtin_amdgcn_update_dpp(0, (int)v, 0x114, 0xf, 0xf, false);
    v += (u32)__builtin_amdgcn_update_dpp(0, (int)v, 0x118, 0xf, 0xf, false);
    v += (u32)__builtin_amdgcn_update_dpp(0, (int)v, 0x142, 0xa, 0xf, false);
    v += (u32)__builtin_amdgcn_update_dpp(0, (int)v, 0x143, 0xc, 0xf, false);
    return v;
}

#define STAMP_PARAM
#define STAMP_ARG
#define STAMP_DECL
#define STAMP() do {} while (0)
#define STAMP_AT(k, val) do {} while (0)
#define STAMP_SET(k) do {} while (0)
#define STAMP_FLUSH(ws) do {} while (0)

__device__ __forceinline__ int unit_typ(u32 n, int cl) { return (int)(((cl ? SC1.typ : SC0.typ) >> (2 * (n >> 5))) & 3); }
__device__ __forceinline__ int unit_rnd(u32 n, int cl) {
    return (int)(((n < 512 ? (cl ? SC1.rnd0 : SC0.rnd0) : (cl ? SC1.rnd1 : SC0.rnd1)) >> (4 * ((n >> 5) & 15))) & 15);
}
__device__ __forceinline__ const char* unit_src(const Params& P, u32 n, int dst, int cl) {
    const int u = (int)(n >> 2) & 7, wv = (int)n & 3, typ = unit_typ(n, cl), rnd = unit_rnd(n, cl);
    return typ == 0 ? P.w + (size_t)(rnd * 64) * NN + dst * NC + u * 256 + wv * 64
                    : P.x + (size_t)((2 * u + typ - 1) * 256 + wv * 64) * KS + rnd * 64;
}
#define UNIT_LOADS \
    "s_cmp_lg_u32 %[typ], 0\n\ts_cbranch_scc1 1f\n\t" \
    "global_load_dword v8, %[vw], %[sb]\n\tv_add_u32 %[tmp], 0x4000, %[vw]\n\t" \
    "global_load_dword v9, %[tmp], %[sb]\n\tv_add_u32 %[tmp], 0x4000, %[tmp]\n\t" \
    "global_load_dword v10, %[tmp], %[sb]\n\tv_add_u32 %[tmp], 0x4000, %[tmp]\n\t" \
    "global_load_dword v11, %[tmp], %[sb]\n\tv_add_u32 %[tmp], 0x4000, %[tmp]\n\t" \
    "global_load_dword v12, %[tmp], %[sb]\n\tv_add_u32 %[tmp], 0x4000, %[tmp]\n\t" \
    "global_load_dword v13, %[tmp], %[sb]\n\tv_add_u32 %[tmp], 0x4000, %[tmp]\n\t" \
    "global_load_dword v14, %[tmp], %[sb]\n\tv_add_u32 %[tmp], 0x4000, %[tmp]\n\t" \
    "global_load_dword v15, %[tmp], %[sb]\n\tv_add_u32 %[tmp], 0x4000, %[tmp]\n\t" \
    "global_load_dword v16, %[tmp], %[sb]\n\tv_add_u32 %[tmp], 0x4000, %[tmp]\n\t" \
    "global_load_dword v17, %[tmp], %[sb]\n\tv_add_u32 %[tmp], 0x4000, %[tmp]\n\t" \
    "global_load_dword v18, %[tmp], %[sb]\n\tv_add_u32 %[tmp], 0x4000, %[tmp]\n\t" \
    "global_load_dword v19, %[tmp], %[sb]\n\tv_add_u32 %[tmp], 0x4000, %[tmp]\n\t" \
    "global_load_dword v20, %[tmp], %[sb]\n\tv_add_u32 %[tmp], 0x4000, %[tmp]\n\t" \
    "global_load_dword v21, %[tmp], %[sb]\n\tv_add_u32 %[tmp], 0x4000, %[tmp]\n\t" \
    "global_load_dword v22, %[tmp], %[sb]\n\tv_add_u32 %[tmp], 0x4000, %[tmp]\n\t" \
    "global_load_dword v23, %[tmp], %[sb]\n\ts_branch 2f\n" \
    "1:\n\ts_cmp_eq_u32 %[typ], 3\n\ts_cbranch_scc1 2f\n\t" \
    "global_load_dwordx4 v[8:11], %[vx], %[sb]\n\tv_add_u32 %[tmp], 0x2000, %[vx]\n\t" \
    "global_load_dwordx4 v[12:15], %[tmp], %[sb]\n\tv_add_u32 %[tmp], 0x2000, %[tmp]\n\t" \
    "global_load_dwordx4 v[16:19], %[tmp], %[sb]\n\tv_add_u32 %[tmp], 0x2000, %[tmp]\n\t" \
    "global_load_dwordx4 v[20:23], %[tmp], %[sb]\n" \
    "2:\n\t"

__device__ __forceinline__ u32 pusher(const Params& P, int dst, int widx, int lane, u32* callcnt STAMP_PARAM) {
    char* wsd = P.ws[dst];
    const int rel = (P.rank - dst) & 7, cl = rel >> 2;
    u32* cnt = (u32*)(P.ws[P.rank] + OFF_CLAIM + (size_t)dst * 128);
    u32* flags = (u32*)(wsd + OFF_INFLAG + (size_t)P.rank * (24 * 8 * 128));
    u32* gate = (u32*)(P.ws[P.rank] + OFF_GATE + (size_t)(dst * 8 + (lane & 7)) * 128);
    const u32 vw = (u32)((lane >> 4) * 16 * NN + (lane & 15) * 4), vx = (u32)((lane >> 2) * KS + (((lane & 3) ^ ((lane >> 5) << 1)) << 4));
    u32 n = (u32)widx, n1 = (u32)widx + NPL;
    u32x4 L0, L1, L2, L3;
    u32 t, tmp;
#define FETCH(counter, nn) asm volatile("s_mov_b64 exec, 1\n\tglobal_atomic_add %[t], %[c], %[one], off sc0\n\ts_mov_b64 exec, -1\n\t" \
        UNIT_LOADS "s_waitcnt vmcnt(0)" \
        : "=&{v[8:11]}"(L0), "=&{v[12:15]}"(L1), "=&{v[16:19]}"(L2), "=&{v[20:23]}"(L3), [t] "=&v"(t), [tmp] "=&v"(tmp) \
        : [c] "v"(counter), [one] "v"(1u), [typ] "s"((nn) < NUNIT ? unit_typ(nn, cl) : 3), [sb] "s"(unit_src(P, (nn) < NUNIT ? (nn) : 0u, dst, cl)), \
          [vw] "v"(vw), [vx] "v"(vx) : "memory", "scc")
    FETCH(callcnt, n);
    const u32 call = (u32)__builtin_amdgcn_readfirstlane((int)t) + 1;
    STAMP();
    const u32 par = call & 1;
    const u32 cbase = (call - 1) * NUNIT - 2 * NPL;
#pragma unroll 1
    while (n < NUNIT) {
        const int u = (int)(n >> 2) & 7, wv = (int)n & 3, typ = unit_typ(n, cl), rnd = unit_rnd(n, cl);
        u32x4 S[4] = {L0, L1, L2, L3};
        char* d0; int dstep2;
        if (typ == 0) {
            const u32x4 T[4] = {L0, L1, L2, L3};
#pragma unroll
            for (int k = 0; k < 4; ++k) {
                const u32 a = T[k].x, b = T[k].y, c = T[k].z, d = T[k].w;
                const u32 t0 = __builtin_amdgcn_perm(b, a, 0x05010400u), t1 = __builtin_amdgcn_perm(b, a, 0x07030602u);
                const u32 t2 = __builtin_amdgcn_perm(d, c, 0x05010400u), t3 = __builtin_amdgcn_perm(d, c, 0x07030602u);
                S[0][k] = __builtin_amdgcn_perm(t2, t0, 0x05040100u); S[1][k] = __builtin_amdgcn_perm(t2, t0, 0x07060302u);
                S[2][k] = __builtin_amdgcn_perm(t3, t1, 0x05040100u); S[3][k] = __builtin_amdgcn_perm(t3, t1, 0x07060302u);
            }
            d0 = wsd + OFF_WS + par * WS_BYTES + (size_t)((rnd * 8 + rel) * 8 + u) * 16384 + wv * 2048;
            dstep2 = 8192;
        } else {
            d0 = wsd + OFF_XS + par * XS_BYTES + (size_t)(((typ - 1) * 64 + rnd * 8 + rel) * 8 + u) * 16384 + wv * 4096;
            dstep2 = 2048;
        }
        d0 += lane * 16;
        st16_sys(d0, S[0]); st16_sys(d0 + 1024, S[1]); st16_sys(d0 + dstep2, S[2]); st16_sys(d0 + dstep2 + 1024, S[3]);
        FETCH(cnt, n1);
        if (lane == 0) __hip_atomic_store(flags + (n >> 2) * 32 + (n & 3), call, __ATOMIC_RELAXED, __HIP_MEMORY_SCOPE_SYSTEM);
        STAMP();
        n = n1;
        n1 = (u32)__builtin_amdgcn_readfirstlane((int)t) - cbase;
        if (n1 == GATE - 1 + 2 * NPL && lane < 8) __hip_atomic_store(gate, call, __ATOMIC_RELAXED, __HIP_MEMORY_SCOPE_SYSTEM);
    }
#undef FETCH
    return call;
}

__device__ __forceinline__ void expander(const Params& P, LAS char* lds, int src, int cb, int first, int last, int wid, int lane, u32 call STAMP_PARAM) {
    char* ws = P.ws[P.rank];
    const float sc = __builtin_fabsf(P.sx[0] * P.sw[0]);
    LAS char* scr = lds + wid * 16384;
    const int wr = wid >> 2, wc = wid & 3, fr = lane & 15, fq = lane >> 4;
    float* orow = P.out + (size_t)(wr * 64 + fq * 4) * NN + src * NC + cb * 256 + wc * 64 + fr * 4;
#pragma unroll 1
    for (int it8 = first; it8 <= last; ++it8) {
        const int h = it8 >> 2, sg = it8 & 3;
        const int wt = (src * 16 + h * 8 + cb) * 8 + wid;
        const u64* flag = (const u64*)(ws + OFF_OFLAG + (size_t)wt * 128);
        u64 fv;
        {
            const u64 t0 = __builtin_amdgcn_s_memrealtime();
            u32 pass = 0;
            for (;;) {
                fv = __hip_atomic_load(flag + sg, __ATOMIC_RELAXED, __HIP_MEMORY_SCOPE_SYSTEM);
                if ((u32)__builtin_amdgcn_readfirstlane((int)(fv >> 32)) >= call) break;
                if (give_up(ws, call, t0, pass)) break;
                __builtin_amdgcn_s_sleep(XSLEEP);
            }
        }
        STAMP();
        const char* slot = ws + OFF_OSLOT + (size_t)(wt * 4 + sg) * OSLOT;
        const char* pc = slot + 256 + lane * 16;
        u32 mask, tmo; u32x4 c0, c1, c2;
        asm volatile("global_load_dword %0, %5, off sc0 sc1\n\t"
                     "global_load_dwordx4 %1, %6, off sc0 sc1\n\t"
                     "global_load_dwordx4 %2, %6, off offset:1024 sc0 sc1\n\t"
                     "global_load_dwordx4 %3, %6, off offset:2048 sc0 sc1\n\t"
                     "global_load_dword %4, %7, off sc1\n\t"
                     "s_waitcnt vmcnt(0)"
                     : "=&v"(mask), "=&v"(c0), "=&v"(c1), "=&v"(c2), "=&v"(tmo) : "v"(slot + lane * 4), "v"(pc), "v"(ws + OFF_TMO) : "memory");
        u32 info = (u32)__builtin_amdgcn_readfirstlane((int)(u32)fv);
        if ((u32)__builtin_amdgcn_readfirstlane((int)tmo) == call) info = 2u << 16;
        const u32 mode = info >> 16;
        u32 n = info & 0xffffu; if (n > 2048u) n = 2048u;
        float* o = orow + (size_t)(h * 256 + (sg >> 1) * 128 + (sg & 1) * 32) * NN;
        if (mode >= 2) {
            const float qn = __builtin_nanf("");
#pragma unroll
            for (int g = 0; g < 8; ++g) *(f32x4*)(o + (size_t)((g >> 2) * 16 + (g & 3)) * NN) = (f32x4){qn, qn, qn, qn};
            continue;
        }
        if (mode == 0) {
            auto unpack = [&](u32x4 c, int q) {
                const u32 m21 = 0x1fffffu;
                LAS f32x2* d = (LAS f32x2*)(scr + q * 24);
                d[0] = (f32x2){(float)(c.x & m21) * sc, (float)(__builtin_amdgcn_alignbit(c.y, c.x, 21) & m21) * sc};
                d[1] = (f32x2){(float)((c.y >> 10) & m21) * sc, (float)(c.z & m21) * sc};
                d[2] = (f32x2){(float)(__builtin_amdgcn_alignbit(c.w, c.z, 21) & m21) * sc, (float)((c.w >> 10) & m21) * sc};
            };
            unpack(c0, lane); unpack(c1, 64 + lane); unpack(c2, 128 + lane);
#pragma unroll 1
            for (u32 it = 3; it * 384u < n; ++it) {
                u32x4 c;
                if (it * 64 + lane < 342u) {
                    asm volatile("global_load_dwordx4 %0, %1, off sc0 sc1\n\ts_waitcnt vmcnt(0)" : "=&v"(c) : "v"(pc + it * 1024) : "memory");
                    unpack(c, it * 64 + lane);
                }
            }
        } else {
            auto unraw = [&](u32x4 c, int q) {
                *(LAS f32x4*)(scr + q * 16) = (f32x4){(float)c.x * sc, (float)c.y * sc, (float)c.z * sc, (float)c.w * sc};
            };
            unraw(c0, lane); unraw(c1, 64 + lane); unraw(c2, 128 + lane);
#pragma unroll 1
            for (u32 it = 3; it * 256u < n; ++it) {
                u32x4 c;
                asm volatile("global_load_dwordx4 %0, %1, off sc0 sc1\n\ts_waitcnt vmcnt(0)" : "=&v"(c) : "v"(pc + it * 1024) : "memory");
                unraw(c, it * 64 + lane);
            }
        }
        u32 incl = wave_incl_scan((u32)__builtin_popcount(mask));
        if (incl > 2048u) incl = 2048u;
        u32 a = incl * 4;
        __builtin_amdgcn_wave_barrier();
        u32 v[32];
#pragma unroll
        for (int e = 31; e >= 0; --e) {
            const int t = __builtin_amdgcn_sbfe((int)mask, e, 1);
            a += (u32)t << 2;
            v[e] = *(LAS u32*)(scr + (a & 0x3fffu)) & (u32)t;
        }
#pragma unroll
        for (int g = 0; g < 8; ++g) {
            const u32x4 ov = (u32x4){v[4 * g], v[4 * g + 1], v[4 * g + 2], v[4 * g + 3]};
            float* op = o + (size_t)((g >> 2) * 16 + (g & 3)) * NN;
            asm volatile("global_store_dwordx4 %0, %1, off nt\n\ts_nop 1" :: "v"(op), "v"(ov) : "memory");
        }
        __builtin_amdgcn_wave_barrier();
    }
}

__device__ __forceinline__ u32 pack_seg(const int (&v)[32], LAS char* scr, int lane) {
    u32 mask = 0; int mx = 0;
#pragma unroll
    for (int e = 0; e < 32; ++e) { mask |= (v[e] > 0 ? 1u : 0u) << e; mx = v[e] > mx ? v[e] : mx; }
    const u32 cnt = (u32)__builtin_popcount(mask);
    const u32 incl = wave_incl_scan(cnt);
    const u32 n = (u32)__builtin_amdgcn_readlane((int)incl, 63);
    const bool raw = __builtin_amdgcn_ballot_w64(mx >= (1 << 21)) != 0;
    u32 a = (incl - cnt) * 4;
    __builtin_amdgcn_wave_barrier();
#pragma unroll
    for (int e = 0; e < 32; ++e) if (v[e] > 0) { *(LAS int*)(scr + a) = v[e]; a += 4; }
    *(LAS u32*)(scr + 12288 + lane * 4) = mask;
    __builtin_amdgcn_wave_barrier();
    return n | (raw ? 1u << 16 : 0u);
}
__device__ __forceinline__ void send_seg(const u32 SPLIT, char* slot, LAS char* scr, u32 info, int lane) {
    const u32 n = info & 0xffffu;
    if (lane < 16) st16_sys(slot + lane * 16, *(const LAS u32x4*)(scr + 12288 + lane * 16));
    if (info < 0x10000u) {
        const u32 nch = (n + 5) / 6;
#pragma unroll 1
        for (u32 q = lane; q < nch; q += 64) {
            const LAS u32x2* s = (const LAS u32x2*)(scr + q * 24);
            const u32x2 p0 = s[0], p1 = s[1], p2 = s[2];
            u32x4 c;
            c.x = p0.x | (p0.y << 21); c.y = (p0.y >> 11) | (p1.x << 10);
            c.z = p1.y | (p2.x << 21); c.w = (p2.x >> 11) | (p2.y << 10);
            st16_sys(slot + 256 + q * 16, c);
            if (q < 64 * SPLIT) WAIT_V(0);
        }
    } else {
        const u32 nch = (n + 3) / 4;
#pragma unroll 1
        for (u32 q = lane; q < nch; q += 64) st16_sys(slot + 256 + q * 16, *(const LAS u32x4*)(scr + q * 16));
    }
}
__device__ __forceinline__ void publish(u64* flag, u32 call, u32 info, int lane) {
    if (lane == 0) __hip_atomic_store(flag, ((u64)call << 32) | info, __ATOMIC_RELAXED, __HIP_MEMORY_SCOPE_SYSTEM);
}
__device__ __forceinline__ void pass_gate(char* ws, size_t off, bool remote, int wid, u32 call) {
    if (wid == 0 && remote) wait_words<GSLEEP>(ws, ws + off, call);
    asm volatile("" ::: "memory");
    BAR;
    asm volatile("" ::: "memory");
}

__device__ __forceinline__ const char* pair_flag_addr(const char* ws, int rank, int Pn, int lane, int cb, int own) {
    const int s = (rank + (Pn & 1) * 4 + ((lane >> 3) & 3)) & 7, i = Pn >> 1;
    const u64 tw = (Pn & 1) ? SC1.qof[0] : SC0.qof[0], tx = (Pn & 1) ? SC1.qof[1] : SC0.qof[1];
    const int qw = (int)((tw >> (8 * i)) & 255), qx = (int)((tx >> (8 * i)) & 255);
    return ws + OFF_INFLAG + ((s * 24 + ((lane & 4) ? qx : qw)) * 8 + ((lane & 4) ? own : cb)) * 128 + (lane & 3) * 4;
}

__device__ __forceinline__ void gemm_full(const Params& P, LAS char* lds, int own, int cb, int wid, int lane, u32 call STAMP_PARAM) {
    char* ws = P.ws[P.rank];
    const int tid = threadIdx.x, wr = wid >> 2, wc = wid & 3, fr = lane & 15, fq = lane >> 4;
    const u32 par = call & 1;
    const char* cA = ws + OFF_XS + par * XS_BYTES + (size_t)own * 16384;
    const char* cB = ws + OFF_WS + par * WS_BYTES + (size_t)cb * 16384;
    const u32 voff = (u32)tid * 16;
    const u32 ldsw = (u32)wid * 1024;
    const int aoff = wr * 4096 + fr * 64 + ((fq ^ ((fr >> 3) << 1)) << 4), boff = wc * 2048 + lane * 16;
    STAMP();
#define SA(b, hh) (((b) * 2 + (hh)) * HTB)
#define SB(b, hh) ((4 + (b) * 2 + (hh)) * HTB)
#define STAGE(bufoff, gbase) do { _Pragma("unroll") for (int _i = 0; _i < 2; ++_i) \
        __builtin_amdgcn_global_load_lds((const GAS u32*)((gbase) + _i * SLOTB + voff), (LAS u32*)(lds + (bufoff) + ldsw + _i * 8192), 16, 0, 17); } while (0)
#define PIDX(Pn) ((int)(ORD >> (4 * (Pn))) & 15)
#define SNAP(Pn) do { if (wid == 0) __builtin_amdgcn_global_load_lds((const GAS u32*)pair_flag_addr(ws, P.rank, PIDX(Pn), lane, cb, own), \
        (LAS u32*)(lds + LDS_SNAP + ((Pn) & 1) * 256), 4, 0, 17); } while (0)
#define LDA(dst, b, hh) do { _Pragma("unroll") for (int m = 0; m < 4; ++m) _Pragma("unroll") for (int k = 0; k < 2; ++k) \
        dst[m][k] = *(const LAS i32x4*)(lds + SA(b, hh) + aoff + m * 1024 + k * 8192); } while (0)
#define LDB(dst, b, hh) do { _Pragma("unroll") for (int n = 0; n < 2; ++n) _Pragma("unroll") for (int k = 0; k < 2; ++k) \
        dst[n][k] = *(const LAS i32x4*)(lds + SB(b, hh) + boff + n * 1024 + k * 8192); } while (0)
#define MMA(ai, bj, At, Bt) do { __builtin_amdgcn_s_setprio(1); _Pragma("unroll") for (int m = 0; m < 4; ++m) _Pragma("unroll") for (int n = 0; n < 2; ++n) \
        _Pragma("unroll") for (int k = 0; k < 2; ++k) acc[ai][bj][m][n] = __builtin_amdgcn_mfma_i32_16x16x64_i8(At[m][k], Bt[n][k], acc[ai][bj][m][n], 0, 0, 0); \
        __builtin_amdgcn_s_setprio(0); } while (0)
    i32x4 acc[2][2][4][2];
#pragma unroll
    for (int a = 0; a < 2; ++a)
#pragma unroll
        for (int b = 0; b < 2; ++b)
#pragma unroll
            for (int m = 0; m < 4; ++m)
#pragma unroll
                for (int n = 0; n < 2; ++n) acc[a][b][m][n] = (i32x4){0, 0, 0, 0};
    i32x4 At[4][2], B0[2][2], B1[2][2];
    int Pc = 0;
    bool pf = false;
#pragma unroll 1
    for (;;) {
        const char* a0 = cA + (size_t)PIDX(Pc) * (4 * SLOTB); const char* b0 = cB + (size_t)PIDX(Pc) * (4 * SLOTB);
        if (!pf) {
            if (wid == 0) wait_words<GSLEEP>(ws, pair_flag_addr(ws, P.rank, PIDX(Pc), lane, cb, own), call);
            asm volatile("" ::: "memory");
            if (wr == 0 || Pc == 0) BAR;
            asm volatile("" ::: "memory");
            STAMP();
            if (Pc < 15) SNAP(Pc + 1);
            STAGE(SB(0, 0), b0); STAGE(SA(0, 0), a0); STAGE(SB(0, 1), b0 + 8192); STAGE(SA(0, 1), a0 + 8192);
            if (wr == 1) BAR;
            WAIT_V(4); BAR;
            STAGE(SB(1, 0), b0 + 2 * SLOTB); STAGE(SA(1, 0), a0 + 2 * SLOTB); STAGE(SB(1, 1), b0 + 2 * SLOTB + 8192);
            WAIT_V(6); BAR;
        }
        STAMP();
        pf = false;
        if (Pc < 15) {
            const u32 sv = *(const LAS u32*)(lds + LDS_SNAP + ((Pc + 1) & 1) * 256 + lane * 4);
            pf = __builtin_amdgcn_ballot_w64(sv < call) == 0;
        }
        const char* a2 = cA + (size_t)PIDX((Pc + 1) & 15) * (4 * SLOTB); const char* b2 = cB + (size_t)PIDX((Pc + 1) & 15) * (4 * SLOTB);
        LDB(B0, 0, 0); SCHED; LDA(At, 0, 0); STAGE(SA(1, 1), a0 + 2 * SLOTB + 8192);
        WAIT_L(8); BAR; WAIT_L(0); MMA(0, 0, At, B0); BAR; SCHED;
        LDB(B1, 0, 1); if (pf) STAGE(SB(0, 0), b2);
        BAR; WAIT_L(0); MMA(0, 1, At, B1); BAR;
        LDA(At, 0, 1); if (pf) STAGE(SA(0, 0), a2);
        BAR; WAIT_L(0); MMA(1, 0, At, B0); BAR; SCHED;
        if (pf) { STAGE(SB(0, 1), b2 + 8192); WAIT_V(6); } else { WAIT_V(0); }
        BAR; MMA(1, 1, At, B1); BAR;
        LDB(B0, 1, 0); SCHED; LDA(At, 1, 0);
        if (pf) { if (Pc < 14) SNAP(Pc + 2); STAGE(SA(0, 1), a2 + 8192); }
        WAIT_L(8); BAR; WAIT_L(0); MMA(0, 0, At, B0); BAR; SCHED;
        LDB(B1, 1, 1); if (pf) STAGE(SB(1, 0), b2 + 2 * SLOTB);
        BAR; WAIT_L(0); MMA(0, 1, At, B1); BAR;
        LDA(At, 1, 1); if (pf) STAGE(SA(1, 0), a2 + 2 * SLOTB);
        BAR; WAIT_L(0); MMA(1, 0, At, B0); BAR; SCHED;
        if (pf) { STAGE(SB(1, 1), b2 + 2 * SLOTB + 8192); WAIT_V(6); }
        BAR; MMA(1, 1, At, B1); BAR;
        if (++Pc == 16) break;
    }
    if (wr == 0) BAR;
    STAMP_SET(40);
    STAMP();

    const bool dead = timed_out(ws, call);
    if (P.sx[0] * P.sw[0] < 0.f) {
#pragma unroll
        for (int a = 0; a < 2; ++a)
#pragma unroll
            for (int b = 0; b < 2; ++b)
#pragma unroll
                for (int m = 0; m < 4; ++m)
#pragma unroll
                    for (int n = 0; n < 2; ++n) acc[a][b][m][n] = -acc[a][b][m][n];
    }
    char* wso = P.ws[own];
    const bool remote = own != P.rank;
    const int wt = (P.rank * 16 + cb) * 8 + wid;
    u64* oflag = (u64*)(wso + OFF_OFLAG + (size_t)wt * 128);
    LAS char* scr = lds + wid * 16384;
    u32 info_prev = 0;
#pragma unroll
    for (int sg = 0; sg < 4; ++sg) {
        int v[32];
#pragma unroll
        for (int e = 0; e < 32; ++e) v[e] = acc[sg >> 1][(e >> 1) & 1][(sg & 1) * 2 + (e >> 4)][e & 1][(e >> 2) & 3];
        const u32 info = pack_seg(v, scr, lane);
        if (sg == 0) {
            pass_gate(ws, OFF_GATE + (size_t)(own * 8 + cb) * 128, remote, wid, call);
            STAMP();
        } else {
            if (sg == G2SEG) {
                u32 t;
                asm volatile("s_mov_b64 exec, 1\n\tglobal_atomic_add %0, %1, %2, off sc0\n\ts_mov_b64 exec, -1\n\ts_waitcnt vmcnt(0)"
                             : "=&v"(t) : "v"(ws + OFF_CLAIM2 + (size_t)own * 128), "v"(1u) : "memory");
                if ((u32)__builtin_amdgcn_readfirstlane((int)t) == (call - 1) * 64 + (G2N - 1) && lane < 8)
                    __hip_atomic_store((u32*)(ws + OFF_GATE2 + (size_t)(own * 8 + lane) * 128), call, __ATOMIC_RELAXED, __HIP_MEMORY_SCOPE_SYSTEM);
            } else {
                WAIT_V(0);
            }
            publish(oflag + (sg - 1), call, info_prev, lane);
        }
        send_seg((ASPLIT >> (2 * sg)) & 3, wso + OFF_OSLOT + (size_t)(wt * 4 + sg) * OSLOT, scr, info, lane);
        info_prev = dead ? (2u << 16) : info;
        STAMP();
    }
    WAIT_V(0);
    publish(oflag + 3, call, info_prev, lane);
    STAMP();
#undef SA
#undef SB
#undef STAGE
#undef SNAP
#undef PIDX
#undef LDA
#undef LDB
#undef MMA
}

__device__ __forceinline__ const char* round_flag_addr(const char* ws, int rank, int r, int lane, int cb, int own, int hb) {
    const int s = (rank + (lane >> 3)) & 7;
    const int qw0 = (int)((SC0.qof[0] >> (8 * r)) & 255), qw1 = (int)((SC1.qof[0] >> (8 * r)) & 255);
    const int qx0 = (int)((SC0.qof[2] >> (8 * r)) & 255), qx1 = (int)((SC1.qof[2] >> (8 * r)) & 255);
    const int q = (lane & 4) ? (lane < 32 ? qx0 : qx1) : (lane < 32 ? qw0 : qw1);
    return ws + OFF_INFLAG + ((s * 24 + q) * 8 + ((lane & 4) ? own : cb)) * 128 + ((lane & 4) ? (lane & 3) : 2 * hb + (lane & 1)) * 4;
}

__device__ __forceinline__ void gemm_half(const Params& P, LAS char* lds, int own, int hb, int cb, int wid, int lane, u32 call STAMP_PARAM) {
    char* ws = P.ws[P.rank];
    const int rg = wid & 3, cg = wid >> 2, fr = lane & 15, fq = lane >> 4;
    const u32 par = call & 1;
    const char* cA = ws + OFF_XS + par * XS_BYTES + 64 * SLOTB + (size_t)own * 16384;
    const char* cB = ws + OFF_WS + par * WS_BYTES + (size_t)cb * 16384;
    const u32 voffA = (u32)threadIdx.x * 16, voffB = (u32)(cg * 8192 + hb * 4096 + rg * 1024 + lane * 16);
    const u32 ldsw = (u32)wid * 1024;
    const u32 aoff = rg * 4096 + fr * 64 + ((fq ^ ((fr >> 3) << 1)) << 4), boff = 16384 + cg * 2048 + lane * 16;
    STAMP();
#define STAGE(pos, slot) do { \
        __builtin_amdgcn_global_load_lds((const GAS u32*)(cA + (size_t)(slot) * SLOTB + voffA), (LAS u32*)(lds + (pos) + ldsw), 16, 0, 17); \
        __builtin_amdgcn_global_load_lds((const GAS u32*)(cA + (size_t)(slot) * SLOTB + 8192 + voffA), (LAS u32*)(lds + (pos) + ldsw + 8192), 16, 0, 17); \
        __builtin_amdgcn_global_load_lds((const GAS u32*)(cB + (size_t)(slot) * SLOTB + voffB), (LAS u32*)(lds + (pos) + ldsw + 16384), 16, 0, 17); } while (0)
    i32x4 acc[4][4];
#pragma unroll
    for (int m = 0; m < 4; ++m)
#pragma unroll
        for (int n = 0; n < 4; ++n) acc[m][n] = (i32x4){0, 0, 0, 0};
    int r = 0;
    bool pf = false;
    u32 pos = 0, prev = 0;
#pragma unroll 1
    for (;;) {
        if (!pf) {
            if (wid == 0) wait_words<GSLEEP>(ws, round_flag_addr(ws, P.rank, r, lane, cb, own, hb), call);
            asm volatile("" ::: "memory");
            if (cg == 0 || r == 0) BAR;
            asm volatile("" ::: "memory");
            STAMP();
#pragma unroll
            for (int i = 0; i < 5; ++i) STAGE(i * RSLOT, r * 8 + i);
            pos = 0; prev = 5 * RSLOT;
            WAIT_V(12); BAR;
            if (cg == 1) BAR;
        }
        STAMP();
#pragma unroll
        for (int i = 0; i < 8; ++i) {
            if (i == 0 && r < 7 && wid == 0)
                __builtin_amdgcn_global_load_lds((const GAS u32*)round_flag_addr(ws, P.rank, r + 1, lane, cb, own, hb), (LAS u32*)(lds + LDS_SNAP), 4, 0, 17);
            if (i == 3) {
                const u32 sv = *(const LAS u32*)(lds + LDS_SNAP + lane * 4);
                pf = r < 7 && __builtin_amdgcn_ballot_w64(sv < call) == 0;
            }
            i32x4 At[4], Bt[4];
#pragma unroll
            for (int n = 0; n < 4; ++n) Bt[n] = *(const LAS i32x4*)(lds + pos + boff + (n >> 1) * 4096 + (n & 1) * 1024);
#pragma unroll
            for (int m = 0; m < 4; ++m) At[m] = *(const LAS i32x4*)(lds + pos + aoff + m * 1024);
            SCHED;
            if (i < 3 || pf) STAGE(prev, r * 8 + i + 5);
            WAIT_L(0);
            if (i == 2) WAIT_V(9);
            else if (i < 3 || pf) WAIT_V(12);
            else if (i == 3) WAIT_V(9);
            else if (i == 4) WAIT_V(6);
            else if (i == 5) WAIT_V(3);
            else WAIT_V(0);
            BAR; SCHED;
            __builtin_amdgcn_s_setprio(1);
#pragma unroll
            for (int m = 0; m < 4; ++m)
#pragma unroll
                for (int n = 0; n < 4; ++n) acc[m][n] = __builtin_amdgcn_mfma_i32_16x16x64_i8(At[m], Bt[n], acc[m][n], 0, 0, 0);
            __builtin_amdgcn_s_setprio(0);
            SCHED; BAR;
            prev = pos; pos = pos == (RING - 1) * RSLOT ? 0 : pos + RSLOT;
        }
        if (++r == 8) break;
    }
#undef STAGE
    if (cg == 0) BAR;
    STAMP_SET(40);
    STAMP();

    const bool dead = timed_out(ws, call);
    if (P.sx[0] * P.sw[0] < 0.f) {
#pragma unroll
        for (int m = 0; m < 4; ++m)
#pragma unroll
            for (int n = 0; n < 4; ++n) acc[m][n] = -acc[m][n];
    }
    char* wso = P.ws[own];
    const int wt = (P.rank * 16 + 8 + cb) * 8 + (rg & 1) * 4 + hb * 2 + cg, sg0 = (rg >> 1) * 2;
    u64* oflag = (u64*)(wso + OFF_OFLAG + (size_t)wt * 128) + sg0;
    LAS char* scr = lds + wid * 16384;
    u32 info_prev = 0;
#pragma unroll
    for (int sg = 0; sg < 2; ++sg) {
        int v[32];
#pragma unroll
        for (int e = 0; e < 32; ++e) v[e] = acc[sg * 2 + (e >> 4)][e & 3][(e >> 2) & 3];
        const u32 info = pack_seg(v, scr, lane);
        if (sg == 0) {
            pass_gate(ws, OFF_GATE2 + (size_t)(own * 8 + cb) * 128, own != P.rank, wid, call);
            STAMP();
        } else {
            WAIT_V(0);
            publish(oflag, call, info_prev, lane);
        }
        send_seg((BSPLIT >> (2 * sg)) & 3, wso + OFF_OSLOT + (size_t)(wt * 4 + sg0 + sg) * OSLOT, scr, info, lane);
        info_prev = dead ? (2u << 16) : info;
        STAMP();
    }
    WAIT_V(0);
    publish(oflag + 1, call, info_prev, lane);
    STAMP_SET(46);
    STAMP();
    STAMP_FLUSH(ws);
}

__global__ __launch_bounds__(NTHR, 2) void fused_kernel(const Params P) {
    extern __shared__ __attribute__((aligned(16))) char smem[];
    LAS char* lds = (LAS char*)smem;
    const int wid = __builtin_amdgcn_readfirstlane((int)(threadIdx.x >> 6)), lane = threadIdx.x & 63;
    const int xcd = blockIdx.x & 7, j = blockIdx.x >> 3;
    char* ws = P.ws[P.rank];
    STAMP_DECL;
    STAMP();
    u32* callcnt = (u32*)(ws + OFF_CNT + (size_t)blockIdx.x * 128) + wid;
    u32 call = 0;
    const int peer = (P.rank + xcd) & 7;
    if (j < 8 && wid < NPW) {
        call = pusher(P, peer, j * NPW + wid, lane, callcnt STAMP_ARG);
    } else {
        if (lane == 0) call = __hip_atomic_fetch_add(callcnt, 1u, __ATOMIC_RELAXED, __HIP_MEMORY_SCOPE_AGENT) + 1;
        call = (u32)__builtin_amdgcn_readfirstlane((int)call);
        if (j >= 16) { gemm_half(P, lds, peer, (j >> 3) & 1, j & 7, wid, lane, call STAMP_ARG); return; }
        if (j >= 8) gemm_full(P, lds, peer, j & 7, wid, lane, call STAMP_ARG);
        else STAMP();
    }
    if (j < 8) {
        STAMP_AT(40, __builtin_amdgcn_s_memrealtime()); STAMP_SET(41);
        if (wid == 0) wait_words<ESLEEP>(ws, ws + OFF_GATE + (size_t)((xcd == 0 ? (peer + 1) & 7 : peer) * 8 + j) * 128, call);
        asm volatile("" ::: "memory");
        BAR;
        asm volatile("" ::: "memory");
    }
    expander(P, lds, j & 7, xcd, j < 8 ? 0 : 6, j < 8 ? 5 : 7, wid, lane, call STAMP_ARG);
    STAMP_SET(49);
    STAMP();
    STAMP_FLUSH(ws);
}

}

extern "C" void kernel_launch(int rank, int nranks, ncclComm_t comm, hipStream_t stream,
                              void* const* d_in, const int* in_sizes, int n_in,
                              void* d_out, int out_size, const int* in_dtypes, int out_dtype,
                              void* const* d_out_all, void* const* d_ws_all, size_t ws_bytes) {
    (void)comm; (void)in_dtypes; (void)out_dtype; (void)d_out_all;
    if (nranks != NR || (unsigned)rank >= (unsigned)NR || n_in != 4 || ws_bytes < WS_NEED ||
        in_sizes[0] != MM * KS || in_sizes[1] != KS * NN || out_size != (MM / NR) * NN) {
        fprintf(stderr, "kernel_launch: unexpected problem shape\n");
        return;
    }
    static bool attr_set[64] = {};
    int dev = 0;
    HIP_CHECK(hipGetDevice(&dev));
    if (!attr_set[dev & 63]) {
        HIP_CHECK(hipFuncSetAttribute((const void*)fused_kernel, hipFuncAttributeMaxDynamicSharedMemorySize, LDS_BYTES));
        attr_set[dev & 63] = true;
    }
    Params P;
    P.x = (const char*)d_in[0]; P.w = (const char*)d_in[1];
    P.sx = (const float*)d_in[2]; P.sw = (const float*)d_in[3];
    P.out = (float*)d_out;
    for (int p = 0; p < NR; ++p) P.ws[p] = (char*)d_ws_all[p];
    P.rank = rank;
    hipLaunchKernelGGL(fused_kernel, dim3(NWG), dim3(NTHR), LDS_BYTES, stream, P);
    HIP_CHECK(hipGetLastError());
}
```

```cpp
#include <hip/hip_runtime.h>
#include <rccl/rccl.h>
#include <cstdio>
#include <cstdint>

#define HIP_CHECK(expr) do { hipError_t _e = (expr); if (_e != hipSuccess) { \
    fprintf(stderr, "HIP error %s at %s:%d\n", hipGetErrorString(_e), __FILE__, __LINE__); \
    return; } } while (0)

namespace {

typedef unsigned u32;
typedef unsigned long long u64;
typedef unsigned u32x2 __attribute__((ext_vector_type(2)));
typedef unsigned u32x4 __attribute__((ext_vector_type(4)));
typedef int i32x4 __attribute__((ext_vector_type(4)));
typedef float f32x2 __attribute__((ext_vector_type(2)));
typedef float f32x4 __attribute__((ext_vector_type(4)));
#define LAS __attribute__((address_space(3)))
#define GAS __attribute__((address_space(1)))

constexpr int NR = 8;
constexpr int MM = 4096, KK = 4096, NN = 16384;
constexpr int KS = KK / NR;
constexpr int NC = NN / NR;
constexpr int NWG = 256, NTHR = 512;

constexpr size_t OFF_CNT = 0;
constexpr size_t OFF_TMO = 32768;
constexpr size_t OFF_CLAIM = 36864;
constexpr size_t OFF_SFLAG = 40960;
constexpr size_t OFF_INFLAG = 524288;
constexpr size_t OFF_OFLAG = 131072;
constexpr size_t OFF_DBG2 = 262144;
constexpr size_t OFF_DBG = 1u << 20;
constexpr size_t OFF_XS = 2u << 20;
constexpr size_t XS_BYTES = 16u << 20;
constexpr size_t OFF_WS = OFF_XS + 2 * XS_BYTES;
constexpr size_t WS_BYTES = 8u << 20;
constexpr size_t OFF_OSLOT = OFF_WS + 2 * WS_BYTES;
constexpr u32 OSLOT = 256 + 8192;
constexpr size_t OFF_EXP = OFF_OSLOT + (size_t)NR * 16 * 8 * 4 * OSLOT;
constexpr size_t EXP_BYTES = (size_t)2560 * 4096;
constexpr size_t WS_NEED = OFF_EXP + 2 * EXP_BYTES;

constexpr size_t SLOTB = 131072;
constexpr int HTB = 16384;
constexpr int RSLOT = 24576, RING = 6;
constexpr int LDS_SNAP = RING * RSLOT;
constexpr int LDS_BYTES = LDS_SNAP + 1024;
constexpr u64 DEADLINE = 30000000ull;

#define SCH_E "w0a0w1a1w2a2w3a3w4a4w5a5w6a6b0w7a7b1b2b3b4b5b6b7"
#define SCH_O "w0a0w1a1w2a2w3a3w4a4w5a5b0w6a6b1w7a7b2b3b4b5b6b7"
struct Sched { u64 qof[3]; u64 typ; u64 rnd0, rnd1; };
constexpr Sched make_sched(const char* p) {
    Sched s{{0, 0, 0}, 0, 0, 0};
    for (int q = 0; q < 24; ++q) {
        const int t = p[2 * q] == 'w' ? 0 : p[2 * q] == 'a' ? 1 : 2, r = p[2 * q + 1] - '0';
        s.qof[t] |= (u64)q << (8 * r);
        s.typ |= (u64)t << (2 * q);
        if (q < 16) s.rnd0 |= (u64)r << (4 * q); else s.rnd1 |= (u64)r << (4 * (q - 16));
    }
    return s;
}
constexpr Sched SC0 = make_sched(SCH_E), SC1 = make_sched(SCH_O);
constexpr u64 make_order() {
    int key[16] = {};
    for (int p = 0; p < 16; ++p) {
        const Sched& s = (p & 1) ? SC1 : SC0;
        const int qw = (int)((s.qof[0] >> (8 * (p >> 1))) & 255), qx = (int)((s.qof[1] >> (8 * (p >> 1))) & 255);
        key[p] = (qw > qx ? qw : qx) * 16 + p;
    }
    u64 o = 0;
    for (int p = 0; p < 16; ++p) {
        int pos = 0;
        for (int k = 0; k < 16; ++k) pos += key[k] < key[p];
        o |= (u64)p << (4 * pos);
    }
    return o;
}
constexpr u64 ORD = make_order();
constexpr int split_round() {
    for (int r = 0; r < 7; ++r)
        if (((SC1.qof[2] >> (8 * r)) & 255) < ((SC0.qof[2] >> (8 * r)) & 255) && ((SC1.qof[2] >> (8 * r + 8)) & 255) == ((SC0.qof[2] >> (8 * r + 8)) & 255)) return r;
    return 8;
}
constexpr int RSPLIT = split_round();
#define NPW 3
#define NPUSH 144
#define NPULL 18
#define TRIG 48
#define PSLEEP 4
#define BDEL 24
#define NSW 8
#define NXW 3
#define PG 680
#define HSLEEP 2
#define QWIN 16
#define STAG 12
#define ESLEEP 32
#define XSLEEP 8
#define GSLEEP 8
constexpr u32 NUNIT = 768;
constexpr u32 NPL = 8 * NPW;
static_assert(NPUSH % NPL == 0 && TRIG >= NPL && TRIG <= NPUSH && NPULL >= 8 && NPULL <= 8 * NXW && NPUSH + 2 * NPULL <= NUNIT, "units");

struct Params {
    const char* x; const char* w; const float* sx; const float* sw;
    float* out;
    char* ws[NR];
    int rank;
};

__device__ __forceinline__ void st16_sys(void* p, u32x4 v) {
    asm volatile("global_store_dwordx4 %0, %1, off sc0 sc1\n\ts_nop 1" :: "v"(p), "v"(v) : "memory");
}
#define WAIT_V(n) asm volatile("s_waitcnt vmcnt(" #n ")" ::: "memory")
#define WAIT_L(n) asm volatile("s_waitcnt lgkmcnt(" #n ")" ::: "memory")
#define BAR __builtin_amdgcn_s_barrier()
#define SCHED __builtin_amdgcn_sched_barrier(0)

__device__ __forceinline__ u32 ld_sys32(const char* p) {
    return __hip_atomic_load((const u32*)p, __ATOMIC_RELAXED, __HIP_MEMORY_SCOPE_SYSTEM);
}
__device__ __forceinline__ bool timed_out(char* ws, u32 call) {
    return __hip_atomic_load((const u32*)(ws + OFF_TMO), __ATOMIC_RELAXED, __HIP_MEMORY_SCOPE_AGENT) == call;
}
__device__ __forceinline__ bool give_up(char* ws, u32 call, u64 t0, u32& pass) {
    if ((++pass & 255u) == 0 && timed_out(ws, call)) return true;
    if (__builtin_amdgcn_s_memrealtime() - t0 > DEADLINE) {
        __hip_atomic_store((u32*)(ws + OFF_TMO), call, __ATOMIC_RELAXED, __HIP_MEMORY_SCOPE_AGENT);
        return true;
    }
    return false;
}
template <int SLEEP> __device__ __forceinline__ void wait_words(char* ws, const char* p, u32 call) {
    const u64 t0 = __builtin_amdgcn_s_memrealtime();
    u32 pass = 0;
    while (__builtin_amdgcn_ballot_w64(ld_sys32(p) < call) != 0) {
        if (give_up(ws, call, t0, pass)) break;
        __builtin_amdgcn_s_sleep(SLEEP);
    }
}
__device__ __forceinline__ u32 wave_incl_scan(u32 v) {
    v += (u32)__builtin_amdgcn_update_dpp(0, (int)v, 0x111, 0xf, 0xf, false);
    v += (u32)__builtin_amdgcn_update_dpp(0, (int)v, 0x112, 0xf, 0xf, false);
    v += (u32)__builtin_amdgcn_update_dpp(0, (int)v, 0x114, 0xf, 0xf, false);
    v += (u32)__builtin_amdgcn_update_dpp(0, (int)v, 0x118, 0xf, 0xf, false);
    v += (u32)__builtin_amdgcn_update_dpp(0, (int)v, 0x142, 0xa, 0xf, false);
    v += (u32)__builtin_amdgcn_update_dpp(0, (int)v, 0x143, 0xc, 0xf, false);
    return v;
}

#define STAMP_PARAM
#define STAMP_ARG
#define STAMP_DECL
#define STAMP() do {} while (0)
#define STAMP_AT(k, val) do {} while (0)
#define STAMP_SET(k) do {} while (0)
#define STAMP_FLUSH(ws) do {} while (0)

__device__ __forceinline__ int unit_typ(u32 n, int cl) { return (int)(((cl ? SC1.typ : SC0.typ) >> (2 * (n >> 5))) & 3); }
__device__ __forceinline__ int unit_rnd(u32 n, int cl) {
    return (int)(((n < 512 ? (cl ? SC1.rnd0 : SC0.rnd0) : (cl ? SC1.rnd1 : SC0.rnd1)) >> (4 * ((n >> 5) & 15))) & 15);
}
__device__ __forceinline__ const char* unit_src(const Params& P, u32 n, int dst, int cl) {
    const int u = (int)(n >> 2) & 7, wv = (int)n & 3, typ = unit_typ(n, cl), rnd = unit_rnd(n, cl);
    return typ == 0 ? P.w + (size_t)(rnd * 64) * NN + dst * NC + u * 256 + wv * 64
                    : P.x + (size_t)((2 * u + typ - 1) * 256 + wv * 64) * KS + rnd * 64;
}
#define UNIT_LOADS \
    "s_cmp_lg_u32 %[typ], 0\n\ts_cbranch_scc1 1f\n\t" \
    "global_load_dword v8, %[vw], %[sb]\n\tv_add_u32 %[tmp], 0x4000, %[vw]\n\t" \
    "global_load_dword v9, %[tmp], %[sb]\n\tv_add_u32 %[tmp], 0x4000, %[tmp]\n\t" \
    "global_load_dword v10, %[tmp], %[sb]\n\tv_add_u32 %[tmp], 0x4000, %[tmp]\n\t" \
    "global_load_dword v11, %[tmp], %[sb]\n\tv_add_u32 %[tmp], 0x4000, %[tmp]\n\t" \
    "global_load_dword v12, %[tmp], %[sb]\n\tv_add_u32 %[tmp], 0x4000, %[tmp]\n\t" \
    "global_load_dword v13, %[tmp], %[sb]\n\tv_add_u32 %[tmp], 0x4000, %[tmp]\n\t" \
    "global_load_dword v14, %[tmp], %[sb]\n\tv_add_u32 %[tmp], 0x4000, %[tmp]\n\t" \
    "global_load_dword v15, %[tmp], %[sb]\n\tv_add_u32 %[tmp], 0x4000, %[tmp]\n\t" \
    "global_load_dword v16, %[tmp], %[sb]\n\tv_add_u32 %[tmp], 0x4000, %[tmp]\n\t" \
    "global_load_dword v17, %[tmp], %[sb]\n\tv_add_u32 %[tmp], 0x4000, %[tmp]\n\t" \
    "global_load_dword v18, %[tmp], %[sb]\n\tv_add_u32 %[tmp], 0x4000, %[tmp]\n\t" \
    "global_load_dword v19, %[tmp], %[sb]\n\tv_add_u32 %[tmp], 0x4000, %[tmp]\n\t" \
    "global_load_dword v20, %[tmp], %[sb]\n\tv_add_u32 %[tmp], 0x4000, %[tmp]\n\t" \
    "global_load_dword v21, %[tmp], %[sb]\n\tv_add_u32 %[tmp], 0x4000, %[tmp]\n\t" \
    "global_load_dword v22, %[tmp], %[sb]\n\tv_add_u32 %[tmp], 0x4000, %[tmp]\n\t" \
    "global_load_dword v23, %[tmp], %[sb]\n\ts_branch 2f\n" \
    "1:\n\ts_cmp_eq_u32 %[typ], 3\n\ts_cbranch_scc1 2f\n\t" \
    "global_load_dwordx4 v[8:11], %[vx], %[sb]\n\tv_add_u32 %[tmp], 0x2000, %[vx]\n\t" \
    "global_load_dwordx4 v[12:15], %[tmp], %[sb]\n\tv_add_u32 %[tmp], 0x2000, %[tmp]\n\t" \
    "global_load_dwordx4 v[16:19], %[tmp], %[sb]\n\tv_add_u32 %[tmp], 0x2000, %[tmp]\n\t" \
    "global_load_dwordx4 v[20:23], %[tmp], %[sb]\n" \
    "2:\n\t"

__device__ __forceinline__ void transpose_w(const u32x4 (&T)[4], u32x4 (&S)[4]) {
#pragma unroll
    for (int k = 0; k < 4; ++k) {
        const u32 a = T[k].x, b = T[k].y, c = T[k].z, d = T[k].w;
        const u32 t0 = __builtin_amdgcn_perm(b, a, 0x05010400u), t1 = __builtin_amdgcn_perm(b, a, 0x07030602u);
        const u32 t2 = __builtin_amdgcn_perm(d, c, 0x05010400u), t3 = __builtin_amdgcn_perm(d, c, 0x07030602u);
        S[0][k] = __builtin_amdgcn_perm(t2, t0, 0x05040100u); S[1][k] = __builtin_amdgcn_perm(t2, t0, 0x07060302u);
        S[2][k] = __builtin_amdgcn_perm(t3, t1, 0x05040100u); S[3][k] = __builtin_amdgcn_perm(t3, t1, 0x07060302u);
    }
}
__device__ __forceinline__ u32 lane_w(int lane) { return (u32)((lane >> 4) * 16 * NN + (lane & 15) * 4); }
__device__ __forceinline__ u32 lane_x(int lane) { return (u32)((lane >> 2) * KS + (((lane & 3) ^ ((lane >> 5) << 1)) << 4)); }

constexpr u32 first_export() {
    u32 r = 0;
    while (r < 8 && ((SC0.qof[0] >> (8 * r)) & 255) * 32 + 32 <= NPUSH && ((SC1.qof[0] >> (8 * r)) & 255) * 32 + 32 <= NPUSH) ++r;
    return r * 256;
}
__device__ __forceinline__ void exporter(const Params& P, int wg, int wid, int lane, u32 call) {
    char* ex = P.ws[P.rank] + OFF_EXP + (call & 1) * EXP_BYTES + lane * 16;
#pragma unroll 1
    for (u32 s = first_export() + (u32)(wg * NSW + wid); wid < NSW && s < 2560; s += 128 * NSW) {
        const u32 u = (s >> 2) & 7, wv = s & 3;
        u32x4 S[4];
        if (s < 2048) {
            const char* g = P.w + (size_t)((s >> 8) * 64) * NN + ((s >> 5) & 7) * NC + u * 256 + wv * 64 + lane_w(lane);
            u32x4 T[4];
#pragma unroll
            for (int k = 0; k < 16; ++k) T[k >> 2][k & 3] = *(const u32*)(g + k * NN);
            transpose_w(T, S);
        } else {
            const char* g = P.x + (size_t)((2 * u + ((s >> 8) & 1)) * 256 + wv * 64) * KS + ((s >> 5) & 7) * 64 + lane_x(lane);
#pragma unroll
            for (int k = 0; k < 4; ++k) S[k] = *(const u32x4*)(g + k * 16 * KS);
        }
#pragma unroll
        for (int k = 0; k < 4; ++k) st16_sys(ex + (size_t)s * 4096 + k * 1024, S[k]);
    }
    WAIT_V(0);
    BAR;
    if (wid == 0 && lane < NR)
        __hip_atomic_store((u32*)(P.ws[lane] + OFF_SFLAG) + P.rank * 128 + wg, call, __ATOMIC_RELAXED, __HIP_MEMORY_SCOPE_SYSTEM);
}

__device__ __forceinline__ char* unit_dst(char* wsd, u32 par, u32 n, int rel, int lane, int& step2) {
    const int u = (int)(n >> 2) & 7, wv = (int)n & 3, typ = unit_typ(n, rel >> 2), rnd = unit_rnd(n, rel >> 2);
    step2 = typ == 0 ? 8192 : 2048;
    return lane * 16 + (typ == 0 ? wsd + OFF_WS + par * WS_BYTES + (size_t)((rnd * 8 + rel) * 8 + u) * 16384 + wv * 2048
                                 : wsd + OFF_XS + par * XS_BYTES + (size_t)(((typ - 1) * 64 + rnd * 8 + rel) * 8 + u) * 16384 + wv * 4096);
}
__device__ __forceinline__ u32* unit_flag(char* wsd, int src, u32 n) { return (u32*)(wsd + OFF_INFLAG) + (src * 192 + (n >> 2)) * 32 + (n & 3); }

__device__ __forceinline__ u32 pusher(const Params& P, int peer, int j, int wid, int lane, u32* callcnt STAMP_PARAM) {
    char* wsp = P.ws[peer];
    const int rel = (P.rank - peer) & 7;
    const u32 vw = lane_w(lane), vx = lane_x(lane);
    u32 n = (u32)(j * NPW + wid);
    u32x4 L0, L1, L2, L3;
    u32 t, tmp;
#define FETCH(head, nn) asm volatile(head UNIT_LOADS "s_waitcnt vmcnt(0)" \
        : "=&{v[8:11]}"(L0), "=&{v[12:15]}"(L1), "=&{v[16:19]}"(L2), "=&{v[20:23]}"(L3), [t] "=&v"(t), [tmp] "=&v"(tmp) \
        : [c] "v"(callcnt), [one] "v"(1u), [typ] "s"((nn) < NPUSH ? unit_typ(nn, rel >> 2) : 3), [sb] "s"(unit_src(P, nn, peer, rel >> 2)), \
          [vw] "v"(vw), [vx] "v"(vx) : "memory", "scc")
    FETCH("s_mov_b64 exec, 1\n\tglobal_atomic_add %[t], %[c], %[one], off sc0\n\ts_mov_b64 exec, -1\n\t", n);
    BAR;
    const u32 call = (u32)__builtin_amdgcn_readfirstlane((int)t) + 1;
    STAMP();
    for (int k = wid; k > 0; --k) __builtin_amdgcn_s_sleep(STAG);
#pragma unroll 1
    for (; n < NPUSH; n += NPL) {
        u32x4 S[4] = {L0, L1, L2, L3};
        if (unit_typ(n, rel >> 2) == 0) {
            const u32x4 T[4] = {L0, L1, L2, L3};
            transpose_w(T, S);
        }
        int step2;
        char* d = unit_dst(wsp, call & 1, n, rel, lane, step2);
        st16_sys(d, S[0]); st16_sys(d + 1024, S[1]); st16_sys(d + step2, S[2]); st16_sys(d + step2 + 1024, S[3]);
        FETCH("", n + NPL);
        if (lane == 0) __hip_atomic_store(unit_flag(wsp, P.rank, n), call, __ATOMIC_RELAXED, __HIP_MEMORY_SCOPE_SYSTEM);
    }
#undef FETCH
    return call;
}

__device__ __forceinline__ u32 puller(const Params& P, volatile LAS u32x2* q, int peer, u32 pidx, int lane, u32 call) {
    char* ws = P.ws[P.rank];
    const int rel = (peer - P.rank) & 7;
    u32* cnt = (u32*)(ws + OFF_CLAIM + (size_t)peer * 128);
    const char* ex = P.ws[peer] + OFF_EXP + (call & 1) * EXP_BYTES;
    const u32 cbase = (call - 1) * (NUNIT - NPUSH - NPULL) - 2 * NPULL - NPUSH;
    u32x4 L0, L1, L2, L3;
    u32 t;
    auto src = [&](u32 m) {
        const u32 typ = (u32)unit_typ(m, rel >> 2), rnd = (u32)unit_rnd(m, rel >> 2);
        return ex + ((typ == 0 ? rnd * 8 + (u32)P.rank : 56 + typ * 8 + rnd) * 32 + (m & 31)) * 4096;
    };
#define FETCH(head, nn) asm volatile(head \
        "global_load_dwordx4 %0, %[vl], %[sb] sc0 sc1\n\tglobal_load_dwordx4 %1, %[vl], %[sb] offset:1024 sc0 sc1\n\t" \
        "global_load_dwordx4 %2, %[vl], %[sb] offset:2048 sc0 sc1\n\tglobal_load_dwordx4 %3, %[vl], %[sb] offset:3072 sc0 sc1\n\ts_waitcnt vmcnt(0)" \
        : "=&v"(L0), "=&v"(L1), "=&v"(L2), "=&v"(L3), [t] "=&v"(t) : [c] "v"(cnt), [one] "v"(1u), [sb] "s"(src(nn)), [vl] "v"(lane * 16) : "memory")
    wait_words<PSLEEP>(ws, (const char*)unit_flag(ws, peer, NPUSH - TRIG + pidx * NPL / NPULL), call);
    while (q[65].x == 0) __builtin_amdgcn_s_sleep(2);
    u32 n = NPUSH + pidx, n1 = n + NPULL;
    FETCH("", n);
#pragma unroll 1
    for (;;) {
        int step2;
        char* d = unit_dst(ws, call & 1, n, rel, lane, step2);
        st16_sys(d, L0); st16_sys(d + 1024, L1); st16_sys(d + step2, L2); st16_sys(d + step2 + 1024, L3);
        if (n1 >= NUNIT) break;
        FETCH("s_mov_b64 exec, 1\n\tglobal_atomic_add %[t], %[c], %[one], off sc0\n\ts_mov_b64 exec, -1\n\t", n1);
        if (lane == 0) __hip_atomic_store(unit_flag(ws, peer, n), call, __ATOMIC_RELAXED, __HIP_MEMORY_SCOPE_SYSTEM);
        n = n1; n1 = (u32)__builtin_amdgcn_readfirstlane((int)t) - cbase;
    }
#undef FETCH
    return n;
}

__device__ __forceinline__ u32 info_n(u32 info) { return min(info & 0xfffu, 2048u); }
__device__ __forceinline__ u32 info_e(u32 info) { return min((info >> 12) & 0xffu, 64u); }
__device__ __forceinline__ u32 info_chunks(u32 info) {
    const u32 n = info_n(info), G = (n + 7) >> 3;
    return 16 + ((info >> 20) ? (n + 3) >> 2 : G + ((G + 3) >> 2) + ((info_e(info) + 1) >> 1));
}
constexpr u32 XIMG = 9216, XSCR = 2 * XIMG, XSTRIDE = XSCR + 8192;
static_assert(NXW * XSTRIDE <= LDS_SNAP && NXW + NPW < 8 && XIMG >= (OSLOT + 1023) / 1024 * 1024, "LDS of the pulling waves");
__device__ __forceinline__ void watcher(const Params& P, volatile LAS u32x2* q, int src, int k, int lane, u32 call STAMP_PARAM) {
    char* ws = P.ws[P.rank];
    const u32 sg = (0xd8e4u >> ((lane >> 3) * 2)) & 3, wt = (u32)((lane >> 5) * 64 + (lane & 7) * 8 + k);
    const u32 ex = 1u << 31 | (u32)(lane & 7) << 26 | ((u32)(lane >> 5) * 8 + (sg >> 1) * 4 + (sg & 1)) << 22, ey = (wt * 4 + sg) * OSLOT << 6;
    const u64* flag = (const u64*)(ws + OFF_OFLAG + (size_t)(src * 128 + wt) * 128) + sg;
    wait_words<GSLEEP>(ws, ws + OFF_SFLAG + (src * 128 + lane) * 4, call);
    wait_words<GSLEEP>(ws, ws + OFF_SFLAG + (src * 128 + 64 + lane) * 4, call);
    q[65] = (u32x2){1, 0};
    STAMP();
    wait_words<ESLEEP>(ws, ws + OFF_INFLAG + (size_t)(src * 192 + PG / 4) * 128, call);
    const u64 t0 = __builtin_amdgcn_s_memrealtime();
    u32 pass = 0, tail = 0;
    bool pend = true;
    for (;;) {
        const u32 head = q[64].x;
        u64 fv = 0;
        if (pend && (u32)lane < head + QWIN) fv = __hip_atomic_load(flag, __ATOMIC_RELAXED, __HIP_MEMORY_SCOPE_SYSTEM);
        if (give_up(ws, call, t0, pass)) fv = ((u64)call << 32) | (2u << 20);
        const bool rdy = pend && (u32)(fv >> 32) >= call;
        const u64 m = __builtin_amdgcn_ballot_w64(rdy);
        if (rdy) {
            const u32 pos = __builtin_amdgcn_mbcnt_hi((u32)(m >> 32), __builtin_amdgcn_mbcnt_lo((u32)m, tail));
            const u32 info = (u32)fv & 0x3fffffu;
            q[pos] = (u32x2){ex | info, ey | info_chunks(info) * 16};
            pend = false;
        }
        tail += (u32)__builtin_popcountll(m);
        if (tail == 64) break;
        __builtin_amdgcn_s_sleep(XSLEEP);
    }
}

__device__ __forceinline__ void expander(const Params& P, LAS char* xl, volatile LAS u32x2* q, int src, int k, u32 tk, int lane, u32 call STAMP_PARAM) {
    const float sc = __builtin_fabsf(P.sx[0] * P.sw[0]);
    LAS char* scr = xl + XSCR;
    const int fr = lane & 15, fq = lane >> 4;
    char* slots = P.ws[src] + OFF_OSLOT + (size_t)P.rank * (512 * OSLOT);
    const float* out = P.out + (size_t)((k >> 2) * 64) * NN + src * NC + (k & 3) * 64;
    auto wait_entry = [&](u32 n) {
        u32x2 e = q[n];
        while (__builtin_amdgcn_readfirstlane((int)e.x) >= 0) { __builtin_amdgcn_s_sleep(2); e = q[n]; }
        return e;
    };
    auto fetch = [&](u32x2 ev, u32 par) {
        const u32 y = (u32)__builtin_amdgcn_readfirstlane((int)ev.y), so = (y >> 14) << 8, bytes = y & 0x3fffu;
        const auto rs = __builtin_amdgcn_make_buffer_rsrc(slots, 0, (int)(so + bytes), 0x00020000);
        LAS char* d = xl + par * XIMG;
        __builtin_amdgcn_raw_ptr_buffer_load_lds(rs, d, 16, lane * 16, (int)so, 0, 17);
        __builtin_amdgcn_raw_ptr_buffer_load_lds(rs, d, 16, lane * 16, (int)so, 1024, 17);
        __builtin_amdgcn_raw_ptr_buffer_load_lds(rs, d, 16, lane * 16, (int)so, 2048, 17);
#pragma unroll 1
        for (u32 o = 3072; o < bytes; o += 1024) __builtin_amdgcn_raw_ptr_buffer_load_lds(rs, d + o, 16, lane * 16, (int)(so + o), 0, 17);
    };
    u32 oo[8];
#pragma unroll
    for (int g = 0; g < 8; ++g) oo[g] = (u32)((fq * 4 + (g >> 2) * 16 + (g & 3)) * NN + fr * 4) * 4;
    u32x2 ev;
    u32 first = 0;
    if (tk * 8 + k < NPULL) {
        const u32 last = puller(P, q, src, tk * 8 + k, lane, call);
        q[66] = (u32x2){1, 0};
        ev = q[tk];
        first = __builtin_amdgcn_readfirstlane((int)ev.x) < 0;
        if (first) fetch(ev, 0);
        if (first && (__builtin_amdgcn_readfirstlane((int)ev.y) & 0x3fff) <= 3072) WAIT_V(3); else WAIT_V(0);
        if (lane == 0) __hip_atomic_store(unit_flag(P.ws[P.rank], src, last), call, __ATOMIC_RELAXED, __HIP_MEMORY_SCOPE_SYSTEM);
    } else {
        while (q[66].x == 0) __builtin_amdgcn_s_sleep(HSLEEP);
    }
    if (!first) { ev = wait_entry(tk); fetch(ev, 0); }
    u32 e = (u32)__builtin_amdgcn_readfirstlane((int)ev.x);
    STAMP();
    if (first) WAIT_V(1); else WAIT_V(0);
    tk += NXW;
    bool pre = true;
    u32 par = 0;
#pragma unroll 1
    for (;;) {
        if (pre) WAIT_V(8); else WAIT_V(0);
        const LAS char* img = xl + par * XIMG;
        const u32 n = info_n(e), mode = (e >> 20) & 3, G = (n + 7) >> 3, nibo = 256 + 16 * G, exo = nibo + ((G + 3) >> 2) * 16;
        ev = q[min(tk, 64u)];
        u32 mask = *(const LAS u32*)(img + lane * 4);
        u32x4 lo[3];
        u32 nb[3];
#pragma unroll
        for (int i = 0; i < 3; ++i) {
            lo[i] = *(const LAS u32x4*)(img + 256 + (i * 64 + lane) * 16);
            nb[i] = *(const LAS u32*)(img + nibo + (i * 64 + lane) * 4);
        }
        const u32x2 ex = *(const LAS u32x2*)(img + exo + lane * 8);
        pre = __builtin_amdgcn_readfirstlane((int)ev.x) < 0;
        if (pre) fetch(ev, par ^ 1);

        auto conv = [&](u32x4 l, u32 b, u32 g) {
            const u32 na = b & 0x0f0f0f0fu, nh = (b >> 4) & 0x0f0f0f0fu;
            auto cv = [&](u32 hi, u32 w, u32 sel) { return (float)__builtin_amdgcn_perm(hi, w, sel) * sc; };
            *(LAS f32x4*)(scr + g * 32) = (f32x4){cv(na, l.x, 0x0c040100u), cv(na, l.x, 0x0c050302u), cv(na, l.y, 0x0c060100u), cv(na, l.y, 0x0c070302u)};
            *(LAS f32x4*)(scr + g * 32 + 16) = (f32x4){cv(nh, l.z, 0x0c040100u), cv(nh, l.z, 0x0c050302u), cv(nh, l.w, 0x0c060100u), cv(nh, l.w, 0x0c070302u)};
        };
        if (mode == 0) {
            conv(lo[0], nb[0], lane);
            conv(lo[1], nb[1], 64 + lane);
            if (G > 128) conv(lo[2], nb[2], 128 + lane);
            if (G > 192) conv(*(const LAS u32x4*)(img + 256 + (192 + lane) * 16), *(const LAS u32*)(img + nibo + (192 + lane) * 4), 192 + lane);
            __builtin_amdgcn_wave_barrier();
            if ((u32)lane < info_e(e)) *(LAS float*)(scr + (ex.x & 2047u) * 4) = (float)ex.y * sc;
        } else if (mode == 1) {
#pragma unroll 1
            for (u32 q0 = 0; q0 * 4 < n; q0 += 64) {
                const u32x4 c = *(const LAS u32x4*)(img + 256 + (q0 + lane) * 16);
                *(LAS f32x4*)(scr + (q0 + lane) * 16) = (f32x4){(float)c.x * sc, (float)c.y * sc, (float)c.z * sc, (float)c.w * sc};
            }
        } else {
            mask = ~0u;
#pragma unroll 1
            for (u32 q0 = 0; q0 < 512; q0 += 64) *(LAS u32x4*)(scr + (q0 + lane) * 16) = (u32x4){0x7fc00000u, 0x7fc00000u, 0x7fc00000u, 0x7fc00000u};
        }
        u32 a = (u32)(size_t)scr + wave_incl_scan((u32)__builtin_popcount(mask)) * 4;
        if (!pre && tk < 64) {
            ev = q[tk];
            pre = __builtin_amdgcn_readfirstlane((int)ev.x) < 0;
            if (pre) fetch(ev, par ^ 1);
        }
        u32 tk2;
        const float* o = out + ((e >> 22) & 15) * (32 * NN) + ((e >> 26) & 7) * 256;
        __builtin_amdgcn_wave_barrier();
#define RD(e, v, t) "v_bfe_i32 v" #t ", %[m], " #e ", 1\n\tv_lshl_add_u32 %[a], v" #t ", 2, %[a]\n\tds_read_b32 v" #v ", %[a]\n\t"
#define AN(v, t) "v_and_b32 v" #v ", v" #v ", v" #t "\n\t"
#define ST(r, g) "global_store_dwordx4 %[o" #g "], v[" r "], %[o] nt\n\t"
        asm volatile("s_mov_b64 exec, 1\n\tds_add_rtn_u32 %[tk], %[hd], %[one]\n\ts_mov_b64 exec, -1\n\t"
                     RD(31, 95, 103) RD(30, 94, 102) RD(29, 93, 101) RD(28, 92, 100) RD(27, 91, 99) RD(26, 90, 98) RD(25, 89, 97) RD(24, 88, 96)
                     RD(23, 87, 111) RD(22, 86, 110) RD(21, 85, 109) RD(20, 84, 108) RD(19, 83, 107) RD(18, 82, 106) RD(17, 81, 105) "s_waitcnt lgkmcnt(7)\n\t"
                     RD(16, 80, 104)
                     AN(95, 103) AN(94, 102) AN(93, 101) AN(92, 100) AN(91, 99) AN(90, 98) AN(89, 97) AN(88, 96) ST("92:95", 7) ST("88:91", 6)
                     RD(15, 79, 103) RD(14, 78, 102) RD(13, 77, 101) RD(12, 76, 100) RD(11, 75, 99) RD(10, 74, 98) RD(9, 73, 97) "s_waitcnt lgkmcnt(7)\n\t"
                     RD(8, 72, 96)
                     AN(87, 111) AN(86, 110) AN(85, 109) AN(84, 108) AN(83, 107) AN(82, 106) AN(81, 105) AN(80, 104) ST("84:87", 5) ST("80:83", 4)
                     RD(7, 71, 111) RD(6, 70, 110) RD(5, 69, 109) RD(4, 68, 108) RD(3, 67, 107) RD(2, 66, 106) RD(1, 65, 105) "s_waitcnt lgkmcnt(7)\n\t"
                     RD(0, 64, 104)
                     AN(79, 103) AN(78, 102) AN(77, 101) AN(76, 100) AN(75, 99) AN(74, 98) AN(73, 97) AN(72, 96) ST("76:79", 3) ST("72:75", 2)
                     "s_waitcnt lgkmcnt(0)\n\t"
                     AN(71, 111) AN(70, 110) AN(69, 109) AN(68, 108) AN(67, 107) AN(66, 106) AN(65, 105) AN(64, 104) ST("68:71", 1) ST("64:67", 0)
                     "s_nop 1"
                     : [a] "+v"(a), [tk] "=&v"(tk2)
                     : [m] "v"(mask), [o] "s"(o), [hd] "v"((u32)(size_t)(q + 64)), [one] "v"(1u), [o0] "v"(oo[0]), [o1] "v"(oo[1]), [o2] "v"(oo[2]),
                       [o3] "v"(oo[3]), [o4] "v"(oo[4]), [o5] "v"(oo[5]), [o6] "v"(oo[6]), [o7] "v"(oo[7])
                     : "memory", "v64", "v65", "v66", "v67", "v68", "v69", "v70", "v71", "v72", "v73", "v74", "v75", "v76", "v77", "v78", "v79",
                       "v80", "v81", "v82", "v83", "v84", "v85", "v86", "v87", "v88", "v89", "v90", "v91", "v92", "v93", "v94", "v95",
                       "v96", "v97", "v98", "v99", "v100", "v101", "v102", "v103", "v104", "v105", "v106", "v107", "v108", "v109", "v110", "v111");
#undef RD
#undef AN
#undef ST
        if (tk >= 64) break;
        if (!pre) {
            ev = wait_entry(tk);
            fetch(ev, par ^ 1);
        }
        e = (u32)__builtin_amdgcn_readfirstlane((int)ev.x); par ^= 1;
        tk = (u32)__builtin_amdgcn_readfirstlane((int)tk2);
    }
}

__device__ __forceinline__ u32 pack_seg(const int (&v)[32], LAS char* scr, int lane) {
    u32 mask = 0;
#pragma unroll
    for (int e = 31; e >= 0; --e) mask = __builtin_amdgcn_alignbit(mask, (u32)v[e], 31);
    mask = ~mask;
    const u32 cnt = (u32)__builtin_popcount(mask);
    const u32 incl = wave_incl_scan(cnt);
    const u32 n = (u32)__builtin_amdgcn_readlane((int)incl, 63);
    u32 a = (u32)(size_t)scr + 256 + (incl - cnt) * 4;
    __builtin_amdgcn_wave_barrier();
#define KEEP(k) "v_cmpx_le_i32 vcc, 0, %" #k "\n\tds_write_b32 %0, %" #k "\n\tv_add_u32 %0, 4, %0\n\ts_mov_b64 exec, -1\n\t"
#pragma unroll
    for (int e = 0; e < 32; e += 8)
        asm volatile(KEEP(1) KEEP(2) KEEP(3) KEEP(4) KEEP(5) KEEP(6) KEEP(7) KEEP(8)
                     : "+v"(a) : "v"(v[e]), "v"(v[e + 1]), "v"(v[e + 2]), "v"(v[e + 3]), "v"(v[e + 4]), "v"(v[e + 5]), "v"(v[e + 6]), "v"(v[e + 7]) : "vcc", "memory");
#undef KEEP
    *(LAS u32*)(scr + lane * 4) = mask;
    *(LAS u32*)(scr + OSLOT + lane * 4) = mask;
    __builtin_amdgcn_wave_barrier();
    const u32 G = (n + 7) >> 3, nibo = OSLOT + 256 + 16 * G, exo = nibo + ((G + 3) >> 2) * 16;
    LAS u32* list = (LAS u32*)(scr + 15360);
    u32 F = 0;
#pragma unroll 1
    for (u32 g0 = 0; g0 < G; g0 += 64) {
        const u32 g = g0 + lane;
        const u32x4 x = *(const LAS u32x4*)(scr + 256 + g * 32), y = *(const LAS u32x4*)(scr + 272 + g * 32);
        const u32x4 lo = {__builtin_amdgcn_perm(x.y, x.x, 0x05040100u), __builtin_amdgcn_perm(x.w, x.z, 0x05040100u),
                          __builtin_amdgcn_perm(y.y, y.x, 0x05040100u), __builtin_amdgcn_perm(y.w, y.z, 0x05040100u)};
        const u32 pa = __builtin_amdgcn_perm(__builtin_amdgcn_perm(x.w, x.z, 0x0602u), __builtin_amdgcn_perm(x.y, x.x, 0x0602u), 0x05040100u);
        const u32 pb = __builtin_amdgcn_perm(__builtin_amdgcn_perm(y.w, y.z, 0x0602u), __builtin_amdgcn_perm(y.y, y.x, 0x0602u), 0x05040100u);
        if (g < G) {
            *(LAS u32x4*)(scr + OSLOT + 256 + g * 16) = lo;
            *(LAS u32*)(scr + nibo + g * 4) = (pa & 0x0f0f0f0fu) | ((pb << 4) & 0xf0f0f0f0u);
        }
        const bool fl = g < G && (x.x | x.y | x.z | x.w | y.x | y.y | y.z | y.w) >= (1u << 20);
        const u64 m = __builtin_amdgcn_ballot_w64(fl);
        if (fl) list[__builtin_amdgcn_mbcnt_hi((u32)(m >> 32), __builtin_amdgcn_mbcnt_lo((u32)m, F))] = g;
        F += (u32)__builtin_popcountll(m);
    }
    __builtin_amdgcn_wave_barrier();
    u32 E = 0;
#pragma unroll 1
    for (u32 f0 = 0; f0 < F && E <= 64; f0 += 8) {
        const u32 f = f0 + (lane >> 3), idx = list[min(f, F - 1)] * 8 + (lane & 7), val = *(const LAS u32*)(scr + 256 + idx * 4);
        const bool ex = f < F && idx < n && val >= (1u << 20);
        const u64 m = __builtin_amdgcn_ballot_w64(ex);
        const u32 pos = __builtin_amdgcn_mbcnt_hi((u32)(m >> 32), __builtin_amdgcn_mbcnt_lo((u32)m, E));
        if (ex && pos < 64) *(LAS u32x2*)(scr + exo + pos * 8) = (u32x2){idx, val};
        E += (u32)__builtin_popcountll(m);
    }
    __builtin_amdgcn_wave_barrier();
    return E > 64 ? n | 1u << 20 : n | E << 12;
}
__device__ __forceinline__ void send_seg(char* slot, u64* flag, LAS char* scr, u32 info, bool dead, u32 call, int lane) {
    const u32 nch = info_chunks(info);
    const LAS char* img = scr + ((info >> 20) ? 0 : OSLOT);
#pragma unroll 1
    for (u32 q = lane; q < nch; q += 64) st16_sys(slot + q * 16, *(const LAS u32x4*)(img + q * 16));
    WAIT_V(0);
    if (lane == 0) __hip_atomic_store(flag, ((u64)call << 32) | (dead ? 2u << 20 : info), __ATOMIC_RELAXED, __HIP_MEMORY_SCOPE_SYSTEM);
}

__device__ __forceinline__ const char* pair_flag_addr(const char* ws, int rank, int Pn, int lane, int cb, int own) {
    const int s = (rank + (Pn & 1) * 4 + ((lane >> 3) & 3)) & 7, i = Pn >> 1;
    const u64 tw = (Pn & 1) ? SC1.qof[0] : SC0.qof[0], tx = (Pn & 1) ? SC1.qof[1] : SC0.qof[1];
    const int qw = (int)((tw >> (8 * i)) & 255), qx = (int)((tx >> (8 * i)) & 255);
    return ws + OFF_INFLAG + ((s * 24 + ((lane & 4) ? qx : qw)) * 8 + ((lane & 4) ? own : cb)) * 128 + (lane & 3) * 4;
}

__device__ __forceinline__ void gemm_full(const Params& P, LAS char* lds, int own, int cb, int wid, int lane, u32 call STAMP_PARAM) {
    char* ws = P.ws[P.rank];
    const int tid = threadIdx.x, wr = wid >> 2, wc = wid & 3, fr = lane & 15, fq = lane >> 4;
    const u32 par = call & 1;
    const char* cA = ws + OFF_XS + par * XS_BYTES + (size_t)own * 16384;
    const char* cB = ws + OFF_WS + par * WS_BYTES + (size_t)cb * 16384;
    const u32 voff = (u32)tid * 16;
    const u32 ldsw = (u32)wid * 1024;
    const int aoff = wr * 4096 + fr * 64 + ((fq ^ ((fr >> 3) << 1)) << 4), boff = wc * 2048 + lane * 16;
    STAMP();
#define SA(b, hh) (((b) * 2 + (hh)) * HTB)
#define SB(b, hh) ((4 + (b) * 2 + (hh)) * HTB)
#define STAGE(bufoff, gbase) do { _Pragma("unroll") for (int _i = 0; _i < 2; ++_i) \
        __builtin_amdgcn_global_load_lds((const GAS u32*)((gbase) + _i * SLOTB + voff), (LAS u32*)(lds + (bufoff) + ldsw + _i * 8192), 16, 0, 17); } while (0)
#define PIDX(Pn) ((int)(ORD >> (4 * (Pn))) & 15)
#define SNAP(Pn) do { if (wid == 0) __builtin_amdgcn_global_load_lds((const GAS u32*)pair_flag_addr(ws, P.rank, PIDX(Pn), lane, cb, own), \
        (LAS u32*)(lds + LDS_SNAP + ((Pn) & 1) * 256), 4, 0, 17); } while (0)
#define LDA(dst, b, hh) do { _Pragma("unroll") for (int m = 0; m < 4; ++m) _Pragma("unroll") for (int k = 0; k < 2; ++k) \
        dst[m][k] = *(const LAS i32x4*)(lds + SA(b, hh) + aoff + m * 1024 + k * 8192); } while (0)
#define LDB(dst, b, hh) do { _Pragma("unroll") for (int n = 0; n < 2; ++n) _Pragma("unroll") for (int k = 0; k < 2; ++k) \
        dst[n][k] = *(const LAS i32x4*)(lds + SB(b, hh) + boff + n * 1024 + k * 8192); } while (0)
#define MMA(ai, bj, At, Bt) do { __builtin_amdgcn_s_setprio(1); _Pragma("unroll") for (int m = 0; m < 4; ++m) _Pragma("unroll") for (int n = 0; n < 2; ++n) \
        _Pragma("unroll") for (int k = 0; k < 2; ++k) acc[ai][bj][m][n] = __builtin_amdgcn_mfma_i32_16x16x64_i8(At[m][k], Bt[n][k], acc[ai][bj][m][n], 0, 0, 0); \
        __builtin_amdgcn_s_setprio(0); } while (0)
    i32x4 acc[2][2][4][2];
#pragma unroll
    for (int a = 0; a < 2; ++a)
#pragma unroll
        for (int b = 0; b < 2; ++b)
#pragma unroll
            for (int m = 0; m < 4; ++m)
#pragma unroll
                for (int n = 0; n < 2; ++n) acc[a][b][m][n] = (i32x4){0, 0, 0, 0};
    i32x4 At[4][2], B0[2][2], B1[2][2];
    int Pc = 0;
    bool pf = false;
#pragma unroll 1
    for (;;) {
        const char* a0 = cA + (size_t)PIDX(Pc) * (4 * SLOTB); const char* b0 = cB + (size_t)PIDX(Pc) * (4 * SLOTB);
        if (!pf) {
            if (wid == 0) wait_words<GSLEEP>(ws, pair_flag_addr(ws, P.rank, PIDX(Pc), lane, cb, own), call);
            asm volatile("" ::: "memory");
            if (wr == 0 || Pc == 0) BAR;
            asm volatile("" ::: "memory");
            STAMP();
            if (Pc < 15) SNAP(Pc + 1);
            STAGE(SB(0, 0), b0); STAGE(SA(0, 0), a0); STAGE(SB(0, 1), b0 + 8192); STAGE(SA(0, 1), a0 + 8192);
            if (wr == 1) BAR;
            WAIT_V(4); BAR;
            STAGE(SB(1, 0), b0 + 2 * SLOTB); STAGE(SA(1, 0), a0 + 2 * SLOTB); STAGE(SB(1, 1), b0 + 2 * SLOTB + 8192);
            WAIT_V(6); BAR;
        }
        STAMP();
        pf = false;
        if (Pc < 15) {
            const u32 sv = *(const LAS u32*)(lds + LDS_SNAP + ((Pc + 1) & 1) * 256 + lane * 4);
            pf = __builtin_amdgcn_ballot_w64(sv < call) == 0;
        }
        const char* a2 = cA + (size_t)PIDX((Pc + 1) & 15) * (4 * SLOTB); const char* b2 = cB + (size_t)PIDX((Pc + 1) & 15) * (4 * SLOTB);
        LDB(B0, 0, 0); SCHED; LDA(At, 0, 0); STAGE(SA(1, 1), a0 + 2 * SLOTB + 8192);
        WAIT_L(8); BAR; WAIT_L(0); MMA(0, 0, At, B0); BAR; SCHED;
        LDB(B1, 0, 1); if (pf) STAGE(SB(0, 0), b2);
        BAR; WAIT_L(0); MMA(0, 1, At, B1); BAR;
        LDA(At, 0, 1); if (pf) STAGE(SA(0, 0), a2);
        BAR; WAIT_L(0); MMA(1, 0, At, B0); BAR; SCHED;
        if (pf) { STAGE(SB(0, 1), b2 + 8192); WAIT_V(6); } else { WAIT_V(0); }
        BAR; MMA(1, 1, At, B1); BAR;
        LDB(B0, 1, 0); SCHED; LDA(At, 1, 0);
        if (pf) { if (Pc < 14) SNAP(Pc + 2); STAGE(SA(0, 1), a2 + 8192); }
        WAIT_L(8); BAR; WAIT_L(0); MMA(0, 0, At, B0); BAR; SCHED;
        LDB(B1, 1, 1); if (pf) STAGE(SB(1, 0), b2 + 2 * SLOTB);
        BAR; WAIT_L(0); MMA(0, 1, At, B1); BAR;
        LDA(At, 1, 1); if (pf) STAGE(SA(1, 0), a2 + 2 * SLOTB);
        BAR; WAIT_L(0); MMA(1, 0, At, B0); BAR; SCHED;
        if (pf) { STAGE(SB(1, 1), b2 + 2 * SLOTB + 8192); WAIT_V(6); }
        BAR; MMA(1, 1, At, B1); BAR;
        if (++Pc == 16) break;
    }
    if (wr == 0) BAR;
    STAMP_SET(40);
    STAMP();

    const bool dead = timed_out(ws, call);
    if (P.sx[0] * P.sw[0] < 0.f) {
#pragma unroll
        for (int a = 0; a < 2; ++a)
#pragma unroll
            for (int b = 0; b < 2; ++b)
#pragma unroll
                for (int m = 0; m < 4; ++m)
#pragma unroll
                    for (int n = 0; n < 2; ++n) acc[a][b][m][n] = -acc[a][b][m][n];
    }
    char* wso = P.ws[own];
    const int wt = (P.rank * 16 + cb) * 8 + wid;
    u64* oflag = (u64*)(wso + OFF_OFLAG + (size_t)wt * 128);
    LAS char* scr = lds + wid * 16384;
#pragma unroll
    for (int sg = 0; sg < 4; ++sg) {
        int v[32];
#pragma unroll
        for (int e = 0; e < 32; ++e) v[e] = acc[sg >> 1][(e >> 1) & 1][(sg & 1) * 2 + (e >> 4)][e & 1][(e >> 2) & 3];
        const u32 info = pack_seg(v, scr, lane);
        STAMP_SET(42 + sg);
        send_seg(ws + OFF_OSLOT + (size_t)(((own * 16 + cb) * 8 + wid) * 4 + sg) * OSLOT, oflag + sg, scr, info, dead, call, lane);
        STAMP();
    }
    STAMP();
#undef SA
#undef SB
#undef STAGE
#undef SNAP
#undef PIDX
#undef LDA
#undef LDB
#undef MMA
}

__device__ __forceinline__ const char* round_flag_addr(const char* ws, int rank, int r, int lane, int cb, int own, int hb) {
    const int s = (rank + (lane >> 3)) & 7;
    const int qw0 = (int)((SC0.qof[0] >> (8 * r)) & 255), qw1 = (int)((SC1.qof[0] >> (8 * r)) & 255);
    const int qx0 = (int)((SC0.qof[2] >> (8 * r)) & 255), qx1 = (int)((SC1.qof[2] >> (8 * r)) & 255);
    const int q = (lane & 4) ? (lane < 32 ? qx0 : qx1) : (lane < 32 ? qw0 : qw1);
    return ws + OFF_INFLAG + ((s * 24 + q) * 8 + ((lane & 4) ? own : cb)) * 128 + ((lane & 4) ? (lane & 3) : 2 * hb + (lane & 1)) * 4;
}

__device__ __forceinline__ void gemm_half(const Params& P, LAS char* lds, int own, int hb, int cb, int wid, int lane, u32 call STAMP_PARAM) {
    char* ws = P.ws[P.rank];
    const int rg = wid & 3, cg = wid >> 2, fr = lane & 15, fq = lane >> 4;
    const u32 par = call & 1;
    const char* cA = ws + OFF_XS + par * XS_BYTES + 64 * SLOTB + (size_t)own * 16384;
    const char* cB = ws + OFF_WS + par * WS_BYTES + (size_t)cb * 16384;
    const u32 voffA = (u32)threadIdx.x * 16, voffB = (u32)(cg * 8192 + hb * 4096 + rg * 1024 + lane * 16);
    const u32 ldsw = (u32)wid * 1024;
    const u32 aoff = rg * 4096 + fr * 64 + ((fq ^ ((fr >> 3) << 1)) << 4), boff = 16384 + cg * 2048 + lane * 16;
    STAMP();
#define STAGE1(k, pos, slot) do { \
        if ((k) < 2) __builtin_amdgcn_global_load_lds((const GAS u32*)(cA + (size_t)(slot) * SLOTB + (k) * 8192 + voffA), (LAS u32*)(lds + (pos) + ldsw + (k) * 8192), 16, 0, 17); \
        else __builtin_amdgcn_global_load_lds((const GAS u32*)(cB + (size_t)(slot) * SLOTB + voffB), (LAS u32*)(lds + (pos) + ldsw + 16384), 16, 0, 17); } while (0)
#define STAGE(pos, slot) do { STAGE1(2, pos, slot); STAGE1(0, pos, slot); STAGE1(1, pos, slot); } while (0)
    i32x4 acc[4][4];
#pragma unroll
    for (int m = 0; m < 4; ++m)
#pragma unroll
        for (int n = 0; n < 4; ++n) acc[m][n] = (i32x4){0, 0, 0, 0};
    int r = 0, part = 4;
    bool pf = false;
    u32 pos = 0, prev = 0;
#pragma unroll 1
    for (;;) {
        if (!pf) {
            const bool split = r == RSPLIT;
            if (wid == 0) wait_words<GSLEEP>(ws, round_flag_addr(ws, P.rank, r, split ? (lane & 31) | part * 8 : lane, cb, own, hb), call);
            asm volatile("" ::: "memory");
            if (cg == 0 || (r == 0 && part == 4)) BAR;
            asm volatile("" ::: "memory");
            STAMP();
#pragma unroll
            for (int i = 0; i < 4; ++i) STAGE(i * RSLOT, r * 8 + (split ? part : 0) + i);
            if (split) {
                WAIT_V(9); BAR;
                if (cg == 1) BAR;
#pragma unroll
                for (int i = 0; i < 4; ++i) {
                    i32x4 At[4], Bt[4];
#pragma unroll
                    for (int n = 0; n < 4; ++n) Bt[n] = *(const LAS i32x4*)(lds + i * RSLOT + boff + (n >> 1) * 4096 + (n & 1) * 1024);
#pragma unroll
                    for (int m = 0; m < 4; ++m) At[m] = *(const LAS i32x4*)(lds + i * RSLOT + aoff + m * 1024);
                    WAIT_L(0);
                    if (i == 0) WAIT_V(6); else if (i == 1) WAIT_V(3); else WAIT_V(0);
                    BAR; SCHED;
                    __builtin_amdgcn_s_setprio(1);
#pragma unroll
                    for (int m = 0; m < 4; ++m)
#pragma unroll
                        for (int n = 0; n < 4; ++n) acc[m][n] = __builtin_amdgcn_mfma_i32_16x16x64_i8(At[m], Bt[n], acc[m][n], 0, 0, 0);
                    __builtin_amdgcn_s_setprio(0);
                    SCHED; BAR;
                }
                if (part == 0) ++r;
                part = 0;
                continue;
            }
            STAGE(4 * RSLOT, r * 8 + 4);
            pos = 0; prev = 5 * RSLOT;
            WAIT_V(12); BAR;
            if (cg == 1) BAR;
        }
        STAMP();
#pragma unroll
        for (int i = 0; i < 8; ++i) {
            if (i == 0 && r < 7 && wid == 0)
                __builtin_amdgcn_global_load_lds((const GAS u32*)round_flag_addr(ws, P.rank, r + 1, lane, cb, own, hb), (LAS u32*)(lds + LDS_SNAP), 4, 0, 17);
            if (i == 3) {
                const u32 sv = *(const LAS u32*)(lds + LDS_SNAP + lane * 4);
                pf = r < 7 && r + 1 != RSPLIT && __builtin_amdgcn_ballot_w64(sv < call) == 0;
            }
            i32x4 At[4], Bt[4];
#pragma unroll
            for (int n = 0; n < 4; ++n) Bt[n] = *(const LAS i32x4*)(lds + pos + boff + (n >> 1) * 4096 + (n & 1) * 1024);
#pragma unroll
            for (int m = 0; m < 4; ++m) At[m] = *(const LAS i32x4*)(lds + pos + aoff + m * 1024);
            SCHED;
            const bool st = i < 3 || pf;
            if (st) STAGE(prev, r * 8 + i + 5);
            WAIT_L(0);
            if (i == 2) WAIT_V(9);
            else if (st) WAIT_V(12);
            else if (i == 3) WAIT_V(9);
            else if (i == 4) WAIT_V(6);
            else if (i == 5) WAIT_V(3);
            else WAIT_V(0);
            BAR; SCHED;
            __builtin_amdgcn_s_setprio(1);
#pragma unroll
            for (int m = 0; m < 4; ++m)
#pragma unroll
                for (int n = 0; n < 4; ++n) acc[m][n] = __builtin_amdgcn_mfma_i32_16x16x64_i8(At[m], Bt[n], acc[m][n], 0, 0, 0);
            __builtin_amdgcn_s_setprio(0);
            SCHED; BAR;
            prev = pos; pos = pos == (RING - 1) * RSLOT ? 0 : pos + RSLOT;
        }
        if (++r == 8) break;
    }
#undef STAGE
#undef STAGE1
    if (cg == 0) BAR;
    STAMP_SET(40);
    STAMP();

    const bool dead = timed_out(ws, call);
    if (P.sx[0] * P.sw[0] < 0.f) {
#pragma unroll
        for (int m = 0; m < 4; ++m)
#pragma unroll
            for (int n = 0; n < 4; ++n) acc[m][n] = -acc[m][n];
    }
    char* wso = P.ws[own];
    const int wt = (P.rank * 16 + 8 + cb) * 8 + (rg & 1) * 4 + hb * 2 + cg, sg0 = (rg >> 1) * 2;
    u64* oflag = (u64*)(wso + OFF_OFLAG + (size_t)wt * 128) + sg0;
    LAS char* scr = lds + wid * 16384;
    if (rg >= 2) __builtin_amdgcn_s_sleep(BDEL);
#pragma unroll
    for (int sg = 0; sg < 2; ++sg) {
        int v[32];
#pragma unroll
        for (int e = 0; e < 32; ++e) v[e] = acc[sg * 2 + (e >> 4)][e & 3][(e >> 2) & 3];
        const u32 info = pack_seg(v, scr, lane);
        if (sg == 0) STAMP();
        send_seg(ws + OFF_OSLOT + (size_t)((wt + (own - P.rank) * 128) * 4 + sg0 + sg) * OSLOT, oflag + sg, scr, info, dead, call, lane);
        STAMP();
    }
    STAMP_SET(46);
    STAMP();
    STAMP_FLUSH(ws);
}

__global__ __launch_bounds__(NTHR, 2) void fused_kernel(const Params P) {
    extern __shared__ __attribute__((aligned(16))) char smem[];
    LAS char* lds = (LAS char*)smem;
    const int wid = __builtin_amdgcn_readfirstlane((int)(threadIdx.x >> 6)), lane = threadIdx.x & 63;
    const int xcd = blockIdx.x & 7, j = blockIdx.x >> 3;
    char* ws = P.ws[P.rank];
    STAMP_DECL;
    STAMP();
    u32* callcnt = (u32*)(ws + OFF_CNT + (size_t)blockIdx.x * 128) + wid;
    u32 call = 0;
    const int peer = xcd;
    volatile LAS u32x2* q = (volatile LAS u32x2*)(lds + LDS_SNAP);
    if (j < 8 && wid < NPW) {
        call = pusher(P, peer, j, wid, lane, callcnt STAMP_ARG);
    } else {
        if (lane == 0) call = __hip_atomic_fetch_add(callcnt, 1u, __ATOMIC_RELAXED, __HIP_MEMORY_SCOPE_AGENT) + 1;
        call = (u32)__builtin_amdgcn_readfirstlane((int)call);
        if (j >= 16) {
            exporter(P, (int)blockIdx.x - 128, wid, lane, call);
            gemm_half(P, lds, xcd, j & 1, (j & 15) >> 1, wid, lane, call STAMP_ARG);
            return;
        }
        if (j >= 8) {
            gemm_full(P, lds, xcd, j & 7, wid, lane, call STAMP_ARG);
            STAMP_FLUSH(ws);
            return;
        }
        STAMP();
        if (wid == NPW) { q[lane] = (u32x2){0, 0}; q[64 + lane] = (u32x2){lane ? 0u : 2u * NXW, 0}; WAIT_L(0); }
        BAR;
        asm volatile("" ::: "memory");
        STAMP_AT(40, __builtin_amdgcn_s_memrealtime()); STAMP_SET(41);
        if (wid == NPW) watcher(P, q, peer, j, lane, call STAMP_ARG);
        else if (wid >= 8 - NXW) expander(P, lds + (7 - wid) * XSTRIDE, q, peer, j, 7 - wid, lane, call STAMP_ARG);
    }
    STAMP_SET(49);
    STAMP();
    STAMP_FLUSH(ws);
}

}

extern "C" void kernel_launch(int rank, int nranks, ncclComm_t comm, hipStream_t stream,
                              void* const* d_in, const int* in_sizes, int n_in,
                              void* d_out, int out_size, const int* in_dtypes, int out_dtype,
                              void* const* d_out_all, void* const* d_ws_all, size_t ws_bytes) {
    (void)comm; (void)in_dtypes; (void)out_dtype; (void)d_out_all;
    if (nranks != NR || (unsigned)rank >= (unsigned)NR || n_in != 4 || ws_bytes < WS_NEED ||
        in_sizes[0] != MM * KS || in_sizes[1] != KS * NN || out_size != (MM / NR) * NN) {
        fprintf(stderr, "kernel_launch: unexpected problem shape\n");
        return;
    }
    static bool attr_set[64] = {};
    int dev = 0;
    HIP_CHECK(hipGetDevice(&dev));
    if (!attr_set[dev & 63]) {
        HIP_CHECK(hipFuncSetAttribute((const void*)fused_kernel, hipFuncAttributeMaxDynamicSharedMemorySize, LDS_BYTES));
        attr_set[dev & 63] = true;
    }
    Params P;
    P.x = (const char*)d_in[0]; P.w = (const char*)d_in[1];
    P.sx = (const float*)d_in[2]; P.sw = (const float*)d_in[3];
    P.out = (float*)d_out;
    for (int p = 0; p < NR; ++p) P.ws[p] = (char*)d_ws_all[p];
    P.rank = rank;
    hipLaunchKernelGGL(fused_kernel, dim3(NWG), dim3(NTHR), LDS_BYTES, stream, P);
    HIP_CHECK(hipGetLastError());
}
```
